```python
import numpy as np

kernel_name = "gpu_gemm_ar_m2048_k2048_n2048_f32_gelu"


ABSMAX_RTOL = 0.001

COMPUTE_DTYPE = "float32"
EPILOGUE = "gelu"
IN_QUANT = None
OUT_QUANT = None
IN_DTYPES = {"x": "float32", "w_mat": "float32"}
WORLD_SIZE = 8

def setup_inputs():

    x = np.random.randn(2048, 2048).astype(np.float32)
    w_mat = (np.random.randn(2048, 2048) / np.sqrt(2048)).astype(np.float32)
    return {"x": x, "w_mat": w_mat}

def _epilogue(y):
    c = 0.7978845608028654
    return (0.5 * y * (1.0 + np.tanh(c * (y + 0.044715 * y ** 3)))).astype(np.float32)

def reference(x, w_mat):
    y = (x @ w_mat).astype(np.float32)
    return _epilogue(y)
```

```cpp
#include <hip/hip_runtime.h>
#include <rccl/rccl.h>
#include <cstdio>

#define HIP_CHECK(expr) do { hipError_t _e = (expr); if (_e != hipSuccess) { \
    fprintf(stderr, "HIP error %s at %s:%d\n", hipGetErrorString(_e), __FILE__, __LINE__); \
    return; } } while (0)

typedef _Float16 h8 __attribute__((ext_vector_type(8)));
typedef float f4 __attribute__((ext_vector_type(4)));
typedef unsigned u4 __attribute__((ext_vector_type(4)));
typedef __attribute__((address_space(1))) unsigned gu32;

constexpr int NR = 8;
constexpr int MM = 2048, NN = 2048, KS = 256;
constexpr int NSL = 8;
constexpr int NPW = 32;
constexpr unsigned PROG_OFF = 0;
constexpr unsigned TMO_OFF = 8192;
constexpr unsigned CNT_OFF = 16384;
constexpr unsigned DATA_OFF = 1u << 20;
constexpr unsigned SRC_BYTES = 2u << 20;
constexpr unsigned W_PART = 1u << 20;
constexpr unsigned SLICE_BYTES = 128u << 10;
constexpr unsigned PAR_BYTES = NR * SRC_BYTES;
constexpr size_t WS_NEED = (size_t)DATA_OFF + 2ull * PAR_BYTES;
constexpr unsigned EV_PER_CALL = 32;
constexpr unsigned long long DEADLINE_TICKS = 50000000ull;
constexpr int AUX_SC1 = 16, AUX_SYS = 17;

struct Params {
  const float* x; const float* w; float* out;
  char* ws[NR];
  int rank;
};

template <int N> __device__ __forceinline__ void wait_vm() { asm volatile("s_waitcnt vmcnt(%0)" ::"i"(N) : "memory"); }

__device__ __forceinline__ h8 cvt8(f4 a, f4 b) {
  h8 r;
  r[0] = (_Float16)a[0]; r[1] = (_Float16)a[1]; r[2] = (_Float16)a[2]; r[3] = (_Float16)a[3];
  r[4] = (_Float16)b[0]; r[5] = (_Float16)b[1]; r[6] = (_Float16)b[2]; r[7] = (_Float16)b[3];
  return r;
}

__device__ __forceinline__ void load_unit(const Params& p, int t, int w, int lane, f4 (&raw)[8]) {
  const int s = t >> 1, kq = lane >> 4, l16 = lane & 15;
  if ((t & 1) == 0) {
    const float* base = p.w + (size_t)(32 * s + 4 * kq) * NN + 64 * w + 4 * l16;
#pragma unroll
    for (int j = 0; j < 8; ++j) raw[j] = *(const f4*)(base + (size_t)(16 * (j >> 2) + (j & 3)) * NN);
  } else {
    const float* base = p.x + (size_t)(64 * w + l16) * KS + 32 * s + 4 * kq;
#pragma unroll
    for (int m = 0; m < 4; ++m) {
      raw[2 * m] = *(const f4*)(base + (size_t)(16 * m) * KS);
      raw[2 * m + 1] = *(const f4*)(base + (size_t)(16 * m) * KS + 16);
    }
  }
}

__device__ __forceinline__ void convert_unit(int t, const f4 (&raw)[8], h8 (&cur)[4]) {
  if ((t & 1) == 0) {
#pragma unroll
    for (int e = 0; e < 4; ++e) {
      h8 r;
#pragma unroll
      for (int j = 0; j < 8; ++j) r[j] = (_Float16)raw[j][e];
      cur[e] = r;
    }
  } else {
#pragma unroll
    for (int m = 0; m < 4; ++m) cur[m] = cvt8(raw[2 * m], raw[2 * m + 1]);
  }
}

__device__ __forceinline__ void pusher(const Params& p, int w, unsigned call, int lane) {
  const unsigned dst_off = DATA_OFF + (call & 1u) * PAR_BYTES + (unsigned)p.rank * SRC_BYTES;
  __amdgpu_buffer_rsrc_t rs[NR];
#pragma unroll
  for (int q = 0; q < NR; ++q)
    rs[q] = __builtin_amdgcn_make_buffer_rsrc((void*)(p.ws[(q + w) & (NR - 1)] + dst_off), 0, (int)SRC_BYTES, 0x00020000);
  char* fp = p.ws[0];
#pragma unroll
  for (int q = 1; q < NR; ++q) fp = (lane == q) ? p.ws[q] : fp;
  gu32* flag = (gu32*)(fp + PROG_OFF + w * 128 + p.rank * 4);
  const unsigned ev0 = call * EV_PER_CALL;
  const int voff = w * 4096 + lane * 16;

  f4 raw[8];
  h8 cur[4];
  load_unit(p, 0, w, lane, raw);
#pragma unroll
  for (int t = 0; t < 2 * NSL; ++t) {
    convert_unit(t, raw, cur);
    if (t + 1 < 2 * NSL) load_unit(p, t + 1, w, lane, raw);
    const int soff = (t >> 1) * SLICE_BYTES + ((t & 1) ? 0 : W_PART);
#pragma unroll
    for (int c = 0; c < 4; ++c) {
#pragma unroll
      for (int q = 0; q < NR; ++q)
        __builtin_amdgcn_raw_buffer_store_b128(__builtin_bit_cast(u4, cur[c]), rs[q], voff + c * 1024, soff, AUX_SYS);
      wait_vm<8>();
      if (c == 0 && t > 0) {
        if (lane < NR) __hip_atomic_store(flag, ev0 + t, __ATOMIC_RELAXED, __HIP_MEMORY_SCOPE_SYSTEM);
      }
    }
  }
  wait_vm<0>();
  if (lane < NR) __hip_atomic_store(flag, ev0 + 2 * NSL, __ATOMIC_RELAXED, __HIP_MEMORY_SCOPE_SYSTEM);
}

__device__ __forceinline__ float gelu_tanh(float y) {
  const float k0 = -2.0f * 0.7978845608028654f * 1.4426950408889634f;
  const float k1 = k0 * 0.044715f;
  float t = __builtin_fmaf(y * y, k1, k0);
  float e = __builtin_amdgcn_exp2f(y * t);
  return y * __builtin_amdgcn_rcpf(1.0f + e);
}

__device__ __forceinline__ bool wait_flags(const gu32* fl, unsigned need, gu32* tmo, unsigned call) {
  unsigned v = __hip_atomic_load(fl, __ATOMIC_RELAXED, __HIP_MEMORY_SCOPE_SYSTEM);
  if (__all(v >= need)) return true;
  const unsigned long long t0 = __builtin_amdgcn_s_memrealtime();
  for (;;) {
    __builtin_amdgcn_s_sleep(4);
    v = __hip_atomic_load(fl, __ATOMIC_RELAXED, __HIP_MEMORY_SCOPE_SYSTEM);
    if (__all(v >= need)) return true;
    if (__hip_atomic_load(tmo, __ATOMIC_RELAXED, __HIP_MEMORY_SCOPE_AGENT) == call ||
        __builtin_amdgcn_s_memrealtime() - t0 > DEADLINE_TICKS) {
      __hip_atomic_store(tmo, call, __ATOMIC_RELAXED, __HIP_MEMORY_SCOPE_AGENT);
      return false;
    }
  }
}

__device__ __forceinline__ void consumer(const Params& p, int u, int v, unsigned call, int lane) {
  char* myws = p.ws[p.rank];
  const __amdgpu_buffer_rsrc_t rs =
      __builtin_amdgcn_make_buffer_rsrc((void*)(myws + DATA_OFF + (call & 1u) * PAR_BYTES), 0, (int)PAR_BYTES, 0x00020000);
  const gu32* fl = (const gu32*)(myws + PROG_OFF + ((lane & 8) ? v : u) * 128 + (lane & 7) * 4);
  gu32* tmo = (gu32*)(myws + TMO_OFF);
  const unsigned need0 = call * EV_PER_CALL + ((lane & 8) ? 1u : 2u);
  const int voffA = u * 4096 + lane * 16;
  const int voffB = W_PART + v * 4096 + lane * 16;

  f4 acc[4][4];
#pragma unroll
  for (int m = 0; m < 4; ++m)
#pragma unroll
    for (int e = 0; e < 4; ++e) acc[m][e] = (f4){0.f, 0.f, 0.f, 0.f};

  bool ok = true;
  constexpr int PD = 4;
  for (int s = 0; s < NSL; ++s) {
    ok = wait_flags(fl, need0 + 2 * s, tmo, call);
    if (!ok) break;
    asm volatile("" ::: "memory");
    u4 a[PD][4], b[PD][4];
    const int sbase = s * SLICE_BYTES;
#pragma unroll
    for (int q = 0; q < PD; ++q) {
#pragma unroll
      for (int m = 0; m < 4; ++m) a[q][m] = __builtin_amdgcn_raw_buffer_load_b128(rs, voffA + m * 1024, sbase + q * SRC_BYTES, AUX_SC1);
#pragma unroll
      for (int e = 0; e < 4; ++e) b[q][e] = __builtin_amdgcn_raw_buffer_load_b128(rs, voffB + e * 1024, sbase + q * SRC_BYTES, AUX_SC1);
    }
#pragma unroll
    for (int q = 0; q < NR; ++q) {
      const int sl = q % PD;
#pragma unroll
      for (int m = 0; m < 4; ++m)
#pragma unroll
        for (int e = 0; e < 4; ++e)
          acc[m][e] = __builtin_amdgcn_mfma_f32_16x16x32_f16(__builtin_bit_cast(h8, a[sl][m]), __builtin_bit_cast(h8, b[sl][e]), acc[m][e], 0, 0, 0);
      if (q + PD < NR) {
#pragma unroll
        for (int m = 0; m < 4; ++m) a[sl][m] = __builtin_amdgcn_raw_buffer_load_b128(rs, voffA + m * 1024, sbase + (q + PD) * SRC_BYTES, AUX_SC1);
#pragma unroll
        for (int e = 0; e < 4; ++e) b[sl][e] = __builtin_amdgcn_raw_buffer_load_b128(rs, voffB + e * 1024, sbase + (q + PD) * SRC_BYTES, AUX_SC1);
      }
    }
  }
  if (__hip_atomic_load(tmo, __ATOMIC_RELAXED, __HIP_MEMORY_SCOPE_AGENT) == call) ok = false;
  float* o = p.out + (size_t)(64 * u + 4 * (lane >> 4)) * NN + 64 * v + 4 * (lane & 15);
  const float qnan = __builtin_nanf("");
#pragma unroll
  for (int m = 0; m < 4; ++m)
#pragma unroll
    for (int r = 0; r < 4; ++r) {
      f4 y;
#pragma unroll
      for (int e = 0; e < 4; ++e) y[e] = ok ? gelu_tanh(acc[m][e][r]) : qnan;
      *(f4*)(o + (size_t)(16 * m + r) * NN) = y;
    }
}

__global__ void __launch_bounds__(320, 2) gemm_ag_gelu(Params p) {
  const int lane = threadIdx.x & 63;
  const int wave = __builtin_amdgcn_readfirstlane(threadIdx.x >> 6);
  const int b = blockIdx.x;
  const bool is_pusher_wg = ((b >> 3) & 7) == 0;
  if (wave == 4 && !is_pusher_wg) return;
  gu32* cnt = (gu32*)(p.ws[p.rank] + CNT_OFF + (b * 8 + wave) * 4);
  const unsigned call = __builtin_amdgcn_readfirstlane(__hip_atomic_load(cnt, __ATOMIC_RELAXED, __HIP_MEMORY_SCOPE_AGENT)) + 1u;
  if (lane == 0) __hip_atomic_store(cnt, call, __ATOMIC_RELAXED, __HIP_MEMORY_SCOPE_AGENT);
  if (wave == 4) {
    pusher(p, ((b >> 6) << 3) | (b & 7), call, lane);
  } else {
    const int xcd = b & 7, l = b >> 3;
    const int ti = 4 * (xcd >> 1) + (l >> 3), tj = 8 * (xcd & 1) + (l & 7);
    consumer(p, 2 * ti + (wave >> 1), 2 * tj + (wave & 1), call, lane);
  }
}

extern "C" void kernel_launch(int rank, int nranks, ncclComm_t comm,
                              hipStream_t stream,
                              void* const* d_in, const int* in_sizes, int n_in,
                              void* d_out, int out_size,
                              const int* in_dtypes, int out_dtype,
                              void* const* d_out_all,
                              void* const* d_ws_all, size_t ws_bytes) {
  if (nranks != NR || (unsigned)rank >= (unsigned)NR || n_in < 2 || ws_bytes < WS_NEED ||
      in_sizes[0] != MM * KS || in_sizes[1] != KS * NN || out_size != MM * NN) {
    fprintf(stderr, "kernel_launch: unexpected problem shape\n");
    return;
  }
  Params p;
  p.x = (const float*)d_in[0];
  p.w = (const float*)d_in[1];
  p.out = (float*)d_out;
  for (int q = 0; q < NR; ++q) p.ws[q] = (char*)d_ws_all[q];
  p.rank = rank;
  gemm_ag_gelu<<<256, 320, 0, stream>>>(p);
  HIP_CHECK(hipGetLastError());
}
```

```cpp
#include <hip/hip_runtime.h>
#include <rccl/rccl.h>
#include <cstdio>

#define HIP_CHECK(expr) do { hipError_t _e = (expr); if (_e != hipSuccess) { \
    fprintf(stderr, "HIP error %s at %s:%d\n", hipGetErrorString(_e), __FILE__, __LINE__); \
    return; } } while (0)

typedef _Float16 h8 __attribute__((ext_vector_type(8)));
typedef float f2 __attribute__((ext_vector_type(2)));
typedef float f4 __attribute__((ext_vector_type(4)));
typedef unsigned u4 __attribute__((ext_vector_type(4)));
typedef __attribute__((address_space(1))) unsigned gu32;

#define PW 1
#define THR 16
#define PDEPTH 4
#define POLLK 0
#define EXPM 0

constexpr int NR = 8;
constexpr int MM = 2048, NN = 2048, KS = 256;
constexpr int NSA = 6;
constexpr int NPWG = 32;
constexpr int NPUSH = NPWG * PW;
constexpr int NSTAT = NSA * 128 / NPUSH;
constexpr int NDYN = 256;
constexpr int NG = (256 - NPWG) / 8;
constexpr int NX = 128 - 4 * NG;
constexpr unsigned FLA_OFF = 0;
constexpr unsigned FLB_OFF = 8192;
constexpr unsigned FLC_OFF = 12288;
constexpr unsigned TMO_OFF = 16384;
constexpr unsigned TKT_OFF = 16384 + 128;
constexpr unsigned CNT_OFF = 20480;
constexpr unsigned DATA_OFF = 1u << 20;
constexpr unsigned SRC_BYTES = 2u << 20;
constexpr unsigned W_PART = 1u << 20;
constexpr unsigned SLICE_BYTES = 128u << 10;
constexpr unsigned PAR_BYTES = NR * SRC_BYTES;
constexpr size_t WS_NEED = (size_t)DATA_OFF + 2ull * PAR_BYTES;
constexpr unsigned EV_PER_CALL = 8;
constexpr unsigned long long DEADLINE_TICKS = 50000000ull;
constexpr int AUX_SC1 = 16, AUX_SYS = 17;
constexpr int OOB = 0x7ff00000;

struct Params {
  const float* x; const float* w; float* out;
  char* myws;
  char* ws2[2 * NR];
  int rank;
};

#define STAMP(i) do {} while (0)

template <int N> __device__ __forceinline__ void wait_vm() { asm volatile("s_waitcnt vmcnt(%0)" ::"i"(N) : "memory"); }

__device__ __forceinline__ __amdgpu_buffer_rsrc_t make_rsrc(const void* p, unsigned bytes) {
  return __builtin_amdgcn_make_buffer_rsrc((void*)p, 0, (int)bytes, 0x00020000);
}

__device__ __forceinline__ gu32* call_counter(char* myws) { return (gu32*)(myws + CNT_OFF + (blockIdx.x * 4 + (threadIdx.x >> 6)) * 4); }
__device__ __forceinline__ unsigned next_call(char* myws) {
  gu32* cnt = call_counter(myws);
  const unsigned call = __builtin_amdgcn_readfirstlane(__hip_atomic_load(cnt, __ATOMIC_RELAXED, __HIP_MEMORY_SCOPE_AGENT)) + 1u;
  __hip_atomic_store(cnt, call, __ATOMIC_RELAXED, __HIP_MEMORY_SCOPE_AGENT);
  return call;
}

struct Item {
  int isw;
  int src;
  int sa;
  int dst;
  int dstr;
  int flag;
  int ev;
};

__device__ __forceinline__ Item item_w(int g, int s, int h) {
  Item it;
  it.isw = 1; it.src = (32 * s * NN + 64 * g + 2 * h) * 4; it.sa = 16 * NN * 4;
  it.dst = W_PART + s * SLICE_BYTES + g * 4096 + h * 2048; it.dstr = 1024;
  return it;
}
__device__ __forceinline__ Item static_item(int w, int j) {
  const int g = w & 31, h = j & 1;
  const int type = PW == 2 ? (w >> 5) : ((j >> 1) & 1), s = PW == 2 ? (j >> 1) : (j >> 2);
  Item it = item_w(g, s, h);
  if (type) {
    it.isw = 0; it.src = ((64 * g + 32 * h) * KS + 32 * s) * 4; it.sa = 16 * KS * 4;
    it.dst = s * SLICE_BYTES + g * 4096 + h * 2048;
  }
  it.flag = h ? (int)FLA_OFF + (type * 32 + g) * 128 : -1; it.ev = s + 1;
  return it;
}
__device__ __forceinline__ Item ticket_item(int t) {
  const int g = t & 31, q = (t >> 5) & 3;
  Item it = item_w(g, NSA + (q >> 1), q & 1);
  it.flag = FLB_OFF + g * 128 + q * 32;
  if (t >= 128) {
    const int r = t - 128 - NX, m = r / NG;
    const int strip = r < 0 ? 4 * NG + t - 128 : 4 * (r - m * NG) + m;
    it.isw = 0; it.src = (16 * strip * KS + 32 * NSA) * 4; it.sa = 32 * 4;
    it.dst = NSA * SLICE_BYTES + strip * 1024; it.dstr = SLICE_BYTES;
    it.flag = FLC_OFF + strip * 32;
  }
  it.ev = 7;
  return it;
}

__device__ __forceinline__ void load_item(__amdgpu_buffer_rsrc_t rx, __amdgpu_buffer_rsrc_t rw, const Item& it, int voff, f2 (&raw)[8]) {
  const int b = it.isw ? 2 * NN * 4 : 16 * 4, c = it.isw ? NN * 4 : 2 * 4;
  const __amdgpu_buffer_rsrc_t r = it.isw ? rw : rx;
#pragma unroll
  for (int j = 0; j < 8; ++j)
    raw[j] = __builtin_bit_cast(f2, __builtin_amdgcn_raw_buffer_load_b64(r, voff, it.src + (j >> 2) * it.sa + ((j >> 1) & 1) * b + (j & 1) * c, 0));
}

__device__ __forceinline__ unsigned pk(float a, float b) {
  typedef _Float16 h2 __attribute__((ext_vector_type(2)));
  h2 r; r[0] = (_Float16)a; r[1] = (_Float16)b;
  return __builtin_bit_cast(unsigned, r);
}

__device__ __forceinline__ void pusher(const Params& p, int w, int lane) {
  const __amdgpu_buffer_rsrc_t rx = make_rsrc(p.x, MM * KS * 4), rw = make_rsrc(p.w, KS * NN * 4);
  const __amdgpu_buffer_rsrc_t rc = make_rsrc(p.myws, DATA_OFF);
  const int kq = lane >> 4, l16 = lane & 15;
  const int vw = (4 * kq * NN + 4 * l16) * 4, vx = (l16 * KS + 4 * kq) * 4;
  f2 raw[8];
  Item nxt = static_item(w, 0);
  load_item(rx, rw, nxt, nxt.isw ? vw : vx, raw);
  gu32* cnt = call_counter(p.myws);
  const unsigned cnt0 = __hip_atomic_load(cnt, __ATOMIC_RELAXED, __HIP_MEMORY_SCOPE_AGENT);
  char* fp = p.ws2[lane & (NR - 1)] - DATA_OFF + p.rank * 4;
  __builtin_amdgcn_sched_barrier(0);
  __amdgpu_buffer_rsrc_t rs[NR];
  char* const* tab = p.ws2 + (w & (NR - 1));
#pragma unroll
  for (int q = 0; q < NR; ++q) rs[q] = make_rsrc(tab[q], 2 * PAR_BYTES);
  const unsigned call = __builtin_amdgcn_readfirstlane(cnt0) + 1u;
  __hip_atomic_store(cnt, call, __ATOMIC_RELAXED, __HIP_MEMORY_SCOPE_AGENT);
  STAMP(20);
  const int dbase = (call & 1u) * PAR_BYTES + p.rank * SRC_BYTES;
  const unsigned ev0 = call * EV_PER_CALL;
  const unsigned tbase = (call - 1u) * (NDYN + NPUSH);
  const int voff = lane * 16;
  const int vtk = lane == 0 ? (int)TKT_OFF : OOB;
  int tk = 0;
  int pflag = -1, pev = 0;

#pragma unroll 1
  for (int j = 0;; ++j) {
    const Item cur = nxt;
    u4 fr[2];
#pragma unroll
    for (int e = 0; e < 2; ++e)
#pragma unroll
      for (int d = 0; d < 4; ++d) {
        const unsigned cw = pk(raw[2 * d][e], raw[2 * d + 1][e]), cx = pk(raw[4 * e + d][0], raw[4 * e + d][1]);
        fr[e][d] = cur.isw ? cw : cx;
      }
    if (j == 0) STAMP(21);
    bool more = true;
    if (j + 1 < NSTAT) {
      nxt = static_item(w, j + 1);
    } else {
      const unsigned t = (unsigned)__builtin_amdgcn_readfirstlane(tk) - tbase;
      more = t < (unsigned)NDYN;
      nxt = ticket_item(more ? (int)t : 0);
    }
    __builtin_amdgcn_sched_barrier(0);
    load_item(rx, rw, nxt, !more ? OOB : nxt.isw ? vw : vx, raw);
    tk = __builtin_amdgcn_raw_ptr_buffer_atomic_add_i32(1, rc, (more && j + 2 >= NSTAT) ? vtk : OOB, 0, 0);
    __builtin_amdgcn_sched_barrier(0);
#pragma unroll
    for (int e = 0; e < 2; ++e) {
#pragma unroll
      for (int q = 0; q < NR; ++q)
        __builtin_amdgcn_raw_buffer_store_b128(fr[e], rs[q], voff, dbase + cur.dst + e * cur.dstr, AUX_SYS);
      if (THR == 8) { if (e == 0) wait_vm<17>(); else wait_vm<9>(); } else wait_vm<25>();
      if (e == THR / 8 - 1 && pflag >= 0) {
        if (lane < NR) __hip_atomic_store((gu32*)(fp + pflag), ev0 + pev, __ATOMIC_RELAXED, __HIP_MEMORY_SCOPE_SYSTEM);
      }
    }
    pflag = cur.flag; pev = cur.ev;
    if (!more) break;
  }
  wait_vm<0>();
  if (lane < NR) __hip_atomic_store((gu32*)(fp + pflag), ev0 + pev, __ATOMIC_RELAXED, __HIP_MEMORY_SCOPE_SYSTEM);
  STAMP(11);
}

__device__ __forceinline__ float gelu_tanh(float y, float one) {
  const float k0 = -2.0f * 0.7978845608028654f * 1.4426950408889634f;
  const float k1 = k0 * 0.044715f;
  float t = __builtin_fmaf(y * y, k1, k0);
  float e = __builtin_amdgcn_exp2f(y * t);
  return y * __builtin_amdgcn_rcpf(one + e);
}

__device__ __forceinline__ unsigned long long poll(const gu32* fl, unsigned need) {
  return __ballot(__hip_atomic_load(fl, __ATOMIC_RELAXED, __HIP_MEMORY_SCOPE_SYSTEM) >= need);
}

__device__ __forceinline__ unsigned long long wait_flags(const gu32* fl, unsigned need, unsigned long long want, gu32* tmo, unsigned call, bool& bad) {
  unsigned long long got = poll(fl, need);
  if ((got & want) != want) {
    const unsigned long long t0 = __builtin_amdgcn_s_memrealtime();
    for (unsigned n = 1;; ++n) {
      __builtin_amdgcn_s_sleep(2);
      got = poll(fl, need);
      if ((got & want) == want) break;
      if ((n & 255u) == 0u &&
          (__builtin_amdgcn_readfirstlane(__hip_atomic_load(tmo, __ATOMIC_RELAXED, __HIP_MEMORY_SCOPE_AGENT)) == call ||
           __builtin_amdgcn_s_memrealtime() - t0 > DEADLINE_TICKS)) {
        __hip_atomic_store(tmo, call, __ATOMIC_RELAXED, __HIP_MEMORY_SCOPE_AGENT);
        bad = true;
        got = ~0ull;
        break;
      }
    }
  }
  asm volatile("" ::: "memory");
  return got;
}

#define MFMA16(a, b, c) __builtin_amdgcn_mfma_f32_16x16x32_f16(__builtin_bit_cast(h8, a), __builtin_bit_cast(h8, b), c, 0, 0, 0)
#define LDF(voff, soff) __builtin_amdgcn_raw_buffer_load_b128(rs, voff, soff, AUX_SC1)

__device__ __forceinline__ void consumer(const Params& p, int u, int v, int lane) {
  char* myws = p.myws;
  const unsigned call = next_call(myws);
  const __amdgpu_buffer_rsrc_t rs = make_rsrc(myws + DATA_OFF + (call & 1u) * PAR_BYTES, PAR_BYTES);
  const bool has5 = u < NX;
  const int ux = has5 ? NG + (u >> 2) : u;
  gu32* tmo = (gu32*)(myws + TMO_OFF);
  const unsigned ev0 = call * EV_PER_CALL;
  bool bad = false;
  auto fs = [&](int i) { return !has5 ? 4 * u + i : (i == 0 ? 4 * NG + u : 4 * u + i - 1); };
  int voffA[5];
#pragma unroll
  for (int i = 0; i < 5; ++i) voffA[i] = fs(i) * 1024 + lane * 16;
  const int voffB = W_PART + v * 4096 + lane * 16;

  f4 acc[5][4];
#pragma unroll
  for (int m = 0; m < 5; ++m)
#pragma unroll
    for (int e = 0; e < 4; ++e) acc[m][e] = (f4){0.f, 0.f, 0.f, 0.f};

  constexpr int PD = PDEPTH;
  const gu32* flb = (const gu32*)(myws + FLB_OFF) + v * 32 + (lane & 31);
  unsigned fvb = 0;
  {
    const gu32* fl = (const gu32*)(myws + FLA_OFF) + (lane & 7) + ((lane & 8) ? v : (lane & 16) ? 32 + ux : 32 + u) * 32;
#pragma unroll 1
    for (int s = 0; s < NSA; ++s) {
      wait_flags(fl, ev0 + s + 1, ~0ull, tmo, call, bad);
      STAMP(1 + s);
      u4 a[PD][5], b[PD][4];
      const int sbase = s * SLICE_BYTES;
#pragma unroll
      for (int q = 0; q < PD; ++q) {
#pragma unroll
        for (int m = 0; m < 4; ++m) {
          a[q][m] = LDF(voffA[m], sbase + q * SRC_BYTES);
          b[q][m] = LDF(voffB + m * 1024, sbase + q * SRC_BYTES);
        }
        if (has5) a[q][4] = LDF(voffA[4], sbase + q * SRC_BYTES);
        __builtin_amdgcn_sched_barrier(0);
      }
#pragma unroll
      for (int q = 0; q < NR; ++q) {
        const int sl = q % PD;
#pragma unroll
        for (int m = 0; m < 4; ++m)
#pragma unroll
          for (int e = 0; e < 4; ++e) acc[m][e] = MFMA16(a[sl][m], b[sl][e], acc[m][e]);
        if (has5) {
#pragma unroll
          for (int e = 0; e < 4; ++e) acc[4][e] = MFMA16(a[sl][4], b[sl][e], acc[4][e]);
        }
        __builtin_amdgcn_sched_barrier(0);
        if (q + PD < NR) {
#pragma unroll
          for (int m = 0; m < 4; ++m) {
            a[sl][m] = LDF(voffA[m], sbase + (q + PD) * SRC_BYTES);
            b[sl][m] = LDF(voffB + m * 1024, sbase + (q + PD) * SRC_BYTES);
          }
          if (has5) a[sl][4] = LDF(voffA[4], sbase + (q + PD) * SRC_BYTES);
          __builtin_amdgcn_sched_barrier(0);
        }
        if (q == NR - 2) fvb = __hip_atomic_load(flb, __ATOMIC_RELAXED, __HIP_MEMORY_SCOPE_SYSTEM);
      }
    }
  }
  const int nst = has5 ? 5 : 4;
  const gu32* flc = lane >= 40 ? flb : (const gu32*)(myws + FLC_OFF) + fs((lane >> 3) < nst ? (lane >> 3) : 0) * 8 + (lane & 7);
  u4 bb[16][4];
  unsigned long long got = __ballot(fvb >= ev0 + 7);
  if ((got & 0xffull) != 0xffull) got = wait_flags(flb, ev0 + 7, 0xffull, tmo, call, bad);
  asm volatile("" ::: "memory");
  got &= ~0ull << 40;
#pragma unroll
  for (int q = 0; q < 4; ++q) {
    const unsigned long long want = 0xffull << (32 + 8 * q);
    if (q > 0 && (got & want) != want) got = wait_flags(flc, ev0 + 7, want, tmo, call, bad);
#pragma unroll
    for (int r = 0; r < NR; ++r)
#pragma unroll
      for (int e = 0; e < 2; ++e)
        bb[2 * r + (q >> 1)][2 * (q & 1) + e] = LDF(voffB + (2 * (q & 1) + e) * 1024, (NSA + (q >> 1)) * SLICE_BYTES + r * SRC_BYTES);
    __builtin_amdgcn_sched_barrier(0);
  }
  STAMP(7);

  const __amdgpu_buffer_rsrc_t ro = make_rsrc(p.out, MM * NN * 4);
  const int voffO = (4 * (lane >> 4) * NN + 64 * v + 4 * (lane & 15)) * 4;
  if ((got & 0xffull) != 0xffull) got = wait_flags(flc, ev0 + 7, 0xffull, tmo, call, bad);
  u4 aa[16];
#pragma unroll
  for (int k = 0; k < 16; ++k) aa[k] = LDF(voffA[0], (NSA + (k & 1)) * SLICE_BYTES + (k >> 1) * SRC_BYTES);
#pragma unroll
  for (int r = 0; r < 4; ++r) __builtin_amdgcn_raw_buffer_store_b32(0u, ro, OOB + 256 * r, 0, 0);
#pragma unroll 1
  for (int i = 0; i < nst; ++i) {
    const bool last = i + 1 == nst;
    const unsigned long long wantn = 0xff00ull << (8 * i);
    const bool pf = !last && (got & wantn) == wantn;
    const int voffN = fs(i + 1) * 1024 + lane * 16;
    const int voffP = pf ? voffN : OOB;
    f4 c[4];
#pragma unroll
    for (int e = 0; e < 4; ++e) c[e] = acc[0][e];
    unsigned fv;
    __builtin_amdgcn_sched_barrier(0);
#pragma unroll
    for (int k = 0; k < 16; ++k) {
#pragma unroll
      for (int e = 0; e < 4; ++e) c[e] = MFMA16(aa[k], bb[k][e], c[e]);
      if (k == POLLK) fv = __hip_atomic_load(flc, __ATOMIC_RELAXED, __HIP_MEMORY_SCOPE_SYSTEM);
      aa[k] = LDF(voffP, (NSA + (k & 1)) * SLICE_BYTES + (k >> 1) * SRC_BYTES);
      __builtin_amdgcn_sched_barrier(0);
    }
#pragma unroll
    for (int m = 0; m < 4; ++m)
#pragma unroll
      for (int e = 0; e < 4; ++e) acc[m][e] = acc[m + 1][e];
    __builtin_amdgcn_sched_barrier(0);
    const int srow = fs(i) * 16 * NN * 4;
    const float one = bad ? __builtin_nanf("") : 1.0f;
#pragma unroll
    for (int r = 0; r < 4; ++r) {
      f4 y;
#pragma unroll
      for (int e = 0; e < 4; ++e) y[e] = gelu_tanh(c[e][r], one);
      __builtin_amdgcn_raw_buffer_store_b128(__builtin_bit_cast(u4, y), ro, voffO + r * NN * 4, srow, AUX_SC1);
    }
    __builtin_amdgcn_sched_barrier(0);
    got |= __ballot(fv >= ev0 + 7);
    asm volatile("" ::: "memory");
    if (last) break;
    if (!pf) {
      if ((got & wantn) != wantn) got = wait_flags(flc, ev0 + 7, wantn, tmo, call, bad);
#pragma unroll
      for (int k = 0; k < 16; ++k) aa[k] = LDF(voffN, (NSA + (k & 1)) * SLICE_BYTES + (k >> 1) * SRC_BYTES);
#pragma unroll
      for (int r = 0; r < 4; ++r) __builtin_amdgcn_raw_buffer_store_b32(0u, ro, OOB + 256 * r, 0, 0);
    }
  }
  STAMP(14);
}

__global__ void __launch_bounds__(256, 1) gemm_ag_gelu(Params p) {
  const int lane = threadIdx.x & 63;
  const int wave = __builtin_amdgcn_readfirstlane(threadIdx.x >> 6);
  const int b = blockIdx.x;
  STAMP(0);
  if (b < NPWG) {
    if (wave < PW) pusher(p, wave * NPWG + b, lane);
  } else {
    const int xcd = b & 7, l = (xcd >> 1) * NG + (b >> 3) - NPWG / 8;
    const int ti = l >> 3, tj = (xcd & 1) * 8 + (l & 7);
    consumer(p, 2 * ti + (wave >> 1), 2 * tj + (wave & 1), lane);
  }
}

extern "C" void kernel_launch(int rank, int nranks, ncclComm_t comm,
                              hipStream_t stream,
                              void* const* d_in, const int* in_sizes, int n_in,
                              void* d_out, int out_size,
                              const int* in_dtypes, int out_dtype,
                              void* const* d_out_all,
                              void* const* d_ws_all, size_t ws_bytes) {
  if (nranks != NR || (unsigned)rank >= (unsigned)NR || n_in < 2 || ws_bytes < WS_NEED ||
      in_sizes[0] != MM * KS || in_sizes[1] != KS * NN || out_size != MM * NN) {
    fprintf(stderr, "kernel_launch: unexpected problem shape\n");
    return;
  }
  Params p;
  p.x = (const float*)d_in[0];
  p.w = (const float*)d_in[1];
  p.out = (float*)d_out;
  for (int q = 0; q < 2 * NR; ++q) p.ws2[q] = (char*)d_ws_all[q & (NR - 1)] + DATA_OFF;
  p.myws = (char*)d_ws_all[rank];
  p.rank = rank;
  gemm_ag_gelu<<<256, 256, 0, stream>>>(p);
  HIP_CHECK(hipGetLastError());
}
```

```cpp
#include <hip/hip_runtime.h>
#include <rccl/rccl.h>
#include <cstdio>

#define HIP_CHECK(expr) do { hipError_t _e = (expr); if (_e != hipSuccess) { \
    fprintf(stderr, "HIP error %s at %s:%d\n", hipGetErrorString(_e), __FILE__, __LINE__); \
    return; } } while (0)

typedef _Float16 h8 __attribute__((ext_vector_type(8)));
typedef float f2 __attribute__((ext_vector_type(2)));
typedef float f4 __attribute__((ext_vector_type(4)));
typedef unsigned u4 __attribute__((ext_vector_type(4)));
typedef __attribute__((address_space(1))) unsigned gu32;

#define PW 1
#define THR 16
#define PDEPTH 4
#define DRY 1
#define POLL_SCOPE __HIP_MEMORY_SCOPE_SYSTEM
#define POLL_BITS "sc0 sc1"
#define EXPM 0
#define OUTAUX 16

constexpr int NR = 8;
constexpr int MM = 2048, NN = 2048, KS = 256;
constexpr int NSA = 6;
constexpr int NPWG = 32;
constexpr int NPUSH = NPWG * PW;
constexpr int TB = NSA * 64, TC = TB + 128, NT = TC + 128;
constexpr int NG = (256 - NPWG) / 8;
constexpr int NX = 128 - 4 * NG;
constexpr unsigned FLA_OFF = 0;
constexpr unsigned FLB_OFF = 16384;
constexpr unsigned FLC_OFF = 20480;
constexpr unsigned TMO_OFF = 24576;
constexpr unsigned TKT_OFF = 24576 + 128;
constexpr unsigned CNT_OFF = 28672;
constexpr unsigned DATA_OFF = 1u << 20;
constexpr unsigned SRC_BYTES = 2u << 20;
constexpr unsigned W_PART = 1u << 20;
constexpr unsigned SLICE_BYTES = 128u << 10;
constexpr unsigned PAR_BYTES = NR * SRC_BYTES;
constexpr size_t WS_NEED = (size_t)DATA_OFF + 2ull * PAR_BYTES;
constexpr unsigned long long DEADLINE_TICKS = 50000000ull;
constexpr int AUX_SC1 = 16, AUX_SYS = 17;
constexpr int OOB = 0x70000000;

struct Params {
  const float* x; const float* w; float* out;
  char* myws;
  char* ws2[2 * NR];
  int rank;
};

#define STAMP(i) do {} while (0)

template <int N> __device__ __forceinline__ void wait_vm() { asm volatile("s_waitcnt vmcnt(%0)" ::"i"(N) : "memory"); }

__device__ __forceinline__ __amdgpu_buffer_rsrc_t make_rsrc(const void* p, unsigned bytes) {
  return __builtin_amdgcn_make_buffer_rsrc((void*)p, 0, (int)bytes, 0x00020000);
}

__device__ __forceinline__ gu32* call_counter(char* myws) { return (gu32*)(myws + CNT_OFF + (blockIdx.x * 4 + (threadIdx.x >> 6)) * 4); }
__device__ __forceinline__ unsigned next_call(char* myws) {
  gu32* cnt = call_counter(myws);
  const unsigned call = __builtin_amdgcn_readfirstlane(__hip_atomic_load(cnt, __ATOMIC_RELAXED, __HIP_MEMORY_SCOPE_AGENT)) + 1u;
  __hip_atomic_store(cnt, call, __ATOMIC_RELAXED, __HIP_MEMORY_SCOPE_AGENT);
  return call;
}

struct Item {
  int isw;
  int src;
  int sa;
  int dst;
  int flag;
};

__device__ __forceinline__ Item make_item(int t, int h) {
  const int g = t & 31;
  int s = t >> 6;
  Item it;
  it.flag = h ? (int)FLA_OFF + (t & 63) * 256 + s * 32 : -1;
  if (t >= TB) { s = NSA + ((t >> 6) & 1); h = (t >> 5) & 1; it.flag = FLB_OFF + g * 128 + ((t >> 5) & 3) * 32; }
  it.isw = 1; it.src = (32 * s * NN + 64 * g + 2 * h) * 4; it.sa = 16 * NN * 4;
  it.dst = W_PART + s * SLICE_BYTES + g * 4096 + h * 2048;
  if (t < TB && (t & 32)) {
    it.isw = 0; it.src = ((64 * g + 32 * h) * KS + 32 * s) * 4; it.sa = 16 * KS * 4;
    it.dst -= W_PART;
  }
  if (t >= TC) {
    const int r = t - TC - NX, m = r / NG;
    const int strip = r < 0 ? 4 * NG + t - TC : 4 * (r - m * NG) + m;
    it.isw = 0; it.src = (16 * strip * KS + 32 * NSA) * 4; it.sa = 32 * 4;
    it.dst = NSA * SLICE_BYTES + strip * 2048;
    it.flag = FLC_OFF + strip * 32;
  }
  return it;
}

__device__ __forceinline__ void load_item(__amdgpu_buffer_rsrc_t rx, __amdgpu_buffer_rsrc_t rw, const Item& it, int voff, f2 (&raw)[8]) {
  const int b = it.isw ? 2 * NN * 4 : 16 * 4, c = it.isw ? NN * 4 : 2 * 4;
  const __amdgpu_buffer_rsrc_t r = it.isw ? rw : rx;
#pragma unroll
  for (int j = 0; j < 8; ++j)
    raw[j] = __builtin_bit_cast(f2, __builtin_amdgcn_raw_buffer_load_b64(r, voff, it.src + (j >> 2) * it.sa + ((j >> 1) & 1) * b + (j & 1) * c, 0));
}

__device__ __forceinline__ unsigned pk(float a, float b) {
  typedef _Float16 h2 __attribute__((ext_vector_type(2)));
  h2 r; r[0] = (_Float16)a; r[1] = (_Float16)b;
  return __builtin_bit_cast(unsigned, r);
}

__device__ __forceinline__ void pusher(const Params& p, int w, int lane) {
  const __amdgpu_buffer_rsrc_t rx = make_rsrc(p.x, MM * KS * 4), rw = make_rsrc(p.w, KS * NN * 4);
  const __amdgpu_buffer_rsrc_t rc = make_rsrc(p.myws, DATA_OFF);
  const int kq = lane >> 4, l16 = lane & 15;
  const int vw = (4 * kq * NN + 4 * l16) * 4, vx = (l16 * KS + 4 * kq) * 4;
  f2 raw[8];
  int t = w;
  Item nxt = make_item(t, 0);
  load_item(rx, rw, nxt, nxt.isw ? vw : vx, raw);
  gu32* cnt = call_counter(p.myws);
  const unsigned cnt0 = __hip_atomic_load(cnt, __ATOMIC_RELAXED, __HIP_MEMORY_SCOPE_AGENT);
  char* fp = p.ws2[lane & (NR - 1)] - DATA_OFF + p.rank * 4;
  __builtin_amdgcn_sched_barrier(0);
  __amdgpu_buffer_rsrc_t rs[NR];
  char* const* tab = p.ws2 + (w & (NR - 1));
#pragma unroll
  for (int q = 0; q < NR; ++q) rs[q] = make_rsrc(tab[q], 2 * PAR_BYTES);
  const unsigned call = __builtin_amdgcn_readfirstlane(cnt0) + 1u;
  __hip_atomic_store(cnt, call, __ATOMIC_RELAXED, __HIP_MEMORY_SCOPE_AGENT);
  STAMP(20);
  const int dbase = (call & 1u) * PAR_BYTES + p.rank * SRC_BYTES;
  const unsigned tbase = (call - 1u) * NT - NPUSH;
  const int voff = lane * 16;
  const int vtk = lane == 0 ? (int)TKT_OFF : OOB;
  int tk = 0;
  int pflag = -1;

#pragma unroll 1
  for (int j = 0;; ++j) {
    const Item cur = nxt;
    u4 fr[2];
#pragma unroll
    for (int e = 0; e < 2; ++e)
#pragma unroll
      for (int d = 0; d < 4; ++d) {
        const unsigned cw = pk(raw[2 * d][e], raw[2 * d + 1][e]), cx = pk(raw[4 * e + d][0], raw[4 * e + d][1]);
        fr[e][d] = cur.isw ? cw : cx;
      }
    if (j == 0) STAMP(21);
    bool more = true;
    if (cur.flag >= 0) {
      const unsigned tn = (unsigned)__builtin_amdgcn_readfirstlane(tk) - tbase;
      more = tn < (unsigned)NT;
      t = more ? (int)tn : 0;
    }
    nxt = make_item(t, cur.flag < 0);
    __builtin_amdgcn_sched_barrier(0);
    load_item(rx, rw, nxt, !more ? OOB : nxt.isw ? vw : vx, raw);
    tk = __builtin_amdgcn_raw_ptr_buffer_atomic_add_i32(1, rc, (more && nxt.flag >= 0) ? vtk : OOB, 0, 0);
    __builtin_amdgcn_sched_barrier(0);
#pragma unroll
    for (int e = 0; e < 2; ++e) {
#pragma unroll
      for (int q = 0; q < NR; ++q)
        __builtin_amdgcn_raw_buffer_store_b128(fr[e], rs[q], voff, dbase + cur.dst + e * 1024, AUX_SYS);
      if (THR == 8) { if (e == 0) wait_vm<17>(); else wait_vm<9>(); } else wait_vm<25>();
      if (e == THR / 8 - 1 && pflag >= 0) {
        if (lane < NR) __hip_atomic_store((gu32*)(fp + pflag), call, __ATOMIC_RELAXED, __HIP_MEMORY_SCOPE_SYSTEM);
      }
    }
    pflag = cur.flag;
    if (!more) break;
  }
  wait_vm<0>();
  if (lane < NR) __hip_atomic_store((gu32*)(fp + pflag), call, __ATOMIC_RELAXED, __HIP_MEMORY_SCOPE_SYSTEM);
  STAMP(11);
}

__device__ __forceinline__ float gelu_tanh(float y, float one) {
  const float k0 = -2.0f * 0.7978845608028654f * 1.4426950408889634f;
  const float k1 = k0 * 0.044715f;
  float t = __builtin_fmaf(y * y, k1, k0);
  float e = __builtin_amdgcn_exp2f(y * t);
  return y * __builtin_amdgcn_rcpf(one + e);
}

__device__ __forceinline__ unsigned long long poll(const gu32* fl, unsigned need) {
  return __ballot(__hip_atomic_load(fl, __ATOMIC_RELAXED, POLL_SCOPE) >= need);
}

__device__ __forceinline__ unsigned long long wait_flags(const gu32* fl, unsigned need, unsigned long long want, gu32* tmo, unsigned call, bool& bad) {
  unsigned long long got = poll(fl, need);
  if ((got & want) != want) {
    const unsigned long long t0 = __builtin_amdgcn_s_memrealtime();
    for (unsigned n = 1;; ++n) {
      __builtin_amdgcn_s_sleep(2);
      got = poll(fl, need);
      if ((got & want) == want) break;
      if ((n & 255u) == 0u &&
          (__builtin_amdgcn_readfirstlane(__hip_atomic_load(tmo, __ATOMIC_RELAXED, __HIP_MEMORY_SCOPE_AGENT)) == call ||
           __builtin_amdgcn_s_memrealtime() - t0 > DEADLINE_TICKS)) {
        __hip_atomic_store(tmo, call, __ATOMIC_RELAXED, __HIP_MEMORY_SCOPE_AGENT);
        bad = true;
        got = ~0ull;
        break;
      }
    }
  }
  asm volatile("" ::: "memory");
  return got;
}

#define MFMA16(a, b, c) __builtin_amdgcn_mfma_f32_16x16x32_f16(__builtin_bit_cast(h8, a), __builtin_bit_cast(h8, b), c, 0, 0, 0)
#define LDAUX 16
#define LDBITS "sc1"
#define LDF(voff, soff) __builtin_amdgcn_raw_buffer_load_b128(rs, voff, soff, LDAUX)

#define ALD(c, dst, voff, soff, imm) asm volatile("buffer_load_dwordx4 %0, %1, %2, %3 offen offset:%4 " LDBITS : c(dst) : "v"(voff), "s"(rq), "s"(soff), "i"(imm) : "memory")
#define APOLL(dst, ptr) asm volatile("global_load_dword %0, %1, off " POLL_BITS : "+v"(dst) : "v"(ptr) : "memory")
#define AWAIT(n, c, x) asm volatile("s_waitcnt vmcnt(%1)" : c(x) : "i"(n) : "memory")
#define FILL_ "buffer_store_dword %[oob], %[oob], %[rq], 0 offen\n\t"
#define LD2_(a, b, s) "buffer_load_dwordx4 %" #a ", %[vo], %[rq], %[" #s "] offen " LDBITS "\n\tbuffer_load_dwordx4 %" #b ", %[vo], %[rq], %[" #s "] offen offset:1024 " LDBITS "\n\t"
#define LD8_ LD2_(0, 1, s0) LD2_(2, 3, s1) LD2_(4, 5, s2) LD2_(6, 7, s3)
#define LD8_OUT(k) "+a"(aa[k]), "+a"(aa[k + 1]), "+a"(aa[k + 2]), "+a"(aa[k + 3]), "+a"(aa[k + 4]), "+a"(aa[k + 5]), "+a"(aa[k + 6]), "+a"(aa[k + 7])
#define LD8_IN(k, voff, cond) [s0] "s"(ROFF(k)), [s1] "s"(ROFF(k + 2)), [s2] "s"(ROFF(k + 4)), [s3] "s"(ROFF(k + 6)), [vo] "v"(voff), [rq] "s"(rq), [oob] "v"(OOB), [c] "s"(__builtin_amdgcn_readfirstlane(cond))
#define ROFF(k) (NSA * SLICE_BYTES + ((k) >> 1) * SRC_BYTES)
#define POLL(ptr) __hip_atomic_load(ptr, __ATOMIC_RELAXED, POLL_SCOPE)

__device__ __forceinline__ void consumer(const Params& p, int u, int v, int lane) {
  char* myws = p.myws;
  const unsigned call = next_call(myws);
  const __amdgpu_buffer_rsrc_t rs = make_rsrc(myws + DATA_OFF + (call & 1u) * PAR_BYTES, PAR_BYTES);
  const __amdgpu_buffer_rsrc_t ro = make_rsrc(p.out, MM * NN * 4);
  const unsigned long long rsb = (unsigned long long)(myws + DATA_OFF + (call & 1u) * PAR_BYTES);
  const u4 rq = {(unsigned)rsb, (unsigned)(rsb >> 32) & 0xffffu, PAR_BYTES, 0x00020000u};
  const bool has5 = u < NX;
  const int sx = has5 ? 4 * NG + u : 4 * u;
  gu32* tmo = (gu32*)(myws + TMO_OFF);
  bool bad = false;
  f4 acc[5][4];
#pragma unroll
  for (int m = 0; m < 5; ++m)
#pragma unroll
    for (int e = 0; e < 4; ++e) acc[m][e] = (f4){0.f, 0.f, 0.f, 0.f};
  constexpr int PD = PDEPTH;
  const gu32* fla = (const gu32*)(myws + FLA_OFF) + (lane & 7) + ((lane & 8) ? v : (lane & 16) ? 32 + (sx >> 2) : 32 + u) * 64;
  const gu32* flb = (const gu32*)(myws + FLB_OFF) + v * 32 + (lane & 31);
  const gu32* flc = lane >= 40 ? flb : (const gu32*)(myws + FLC_OFF) + (lane < 8 ? sx : 4 * u + (lane >> 3) - 1) * 8 + (lane & 7);

#pragma unroll 1
  for (int pass = DRY ? 0 : 1; pass < (EXPM >= 4 ? 3 : 2); ++pass) {
    const bool dry = pass == 0;
    const unsigned need = dry ? 0u : call;
    const int vl = lane * 16 + (dry ? OOB : 0);
    const int voffA = vl + u * 4096, voffX = vl + sx * 1024, voffB = vl + W_PART + v * 4096;
    const int voffO = (4 * (lane >> 4) * NN + 64 * v + 4 * (lane & 15)) * 4 + (dry || (EXPM == 5 && pass == 2) ? OOB : 0);
    unsigned fv = 0;
#pragma unroll 1
    for (int s = pass != 1 ? NSA - 1 : 0; s < NSA; ++s) {
      wait_flags(fla + s * 8, need, ~0ull, tmo, call, bad);
      STAMP(1 + s);
      u4 a[PD][5], b[PD][4];
      const int sbase = s * SLICE_BYTES;
#pragma unroll
      for (int q = 0; q < PD; ++q) {
#pragma unroll
        for (int m = 0; m < 4; ++m) {
          a[q][m] = LDF(voffA + m * 1024, sbase + q * SRC_BYTES);
          b[q][m] = LDF(voffB + m * 1024, sbase + q * SRC_BYTES);
        }
        if (has5) a[q][4] = LDF(voffX, sbase + q * SRC_BYTES);
        __builtin_amdgcn_sched_barrier(0);
      }
#pragma unroll
      for (int q = 0; q < NR; ++q) {
        const int sl = q % PD;
#pragma unroll
        for (int m = 0; m < 4; ++m)
#pragma unroll
          for (int e = 0; e < 4; ++e) acc[m][e] = MFMA16(a[sl][m], b[sl][e], acc[m][e]);
        if (has5) {
#pragma unroll
          for (int e = 0; e < 4; ++e) acc[4][e] = MFMA16(a[sl][4], b[sl][e], acc[4][e]);
        }
        __builtin_amdgcn_sched_barrier(0);
        if (q + PD < NR) {
#pragma unroll
          for (int m = 0; m < 4; ++m) {
            a[sl][m] = LDF(voffA + m * 1024, sbase + (q + PD) * SRC_BYTES);
            b[sl][m] = LDF(voffB + m * 1024, sbase + (q + PD) * SRC_BYTES);
          }
          if (has5) a[sl][4] = LDF(voffX, sbase + (q + PD) * SRC_BYTES);
          __builtin_amdgcn_sched_barrier(0);
        }
        if (q == NR - 2) fv = POLL(flb);
      }
    }
    u4 bb[16][4];
    unsigned long long got = __ballot(fv >= need);
    if ((got & 0xffull) != 0xffull) got = wait_flags(flb, need, 0xffull, tmo, call, bad);
    asm volatile("" ::: "memory");
    got &= ~0ull << 40;
#pragma unroll
    for (int q = 0; q < 4; ++q) {
      const unsigned long long want = 0xffull << (32 + 8 * q);
      if (q > 0 && (got & want) != want) got = wait_flags(flc, need, want, tmo, call, bad);
#pragma unroll
      for (int r = 0; r < NR; ++r)
#pragma unroll
        for (int e = 2 * (q & 1); e < 2 * (q & 1) + 2; ++e) {
          const int k = 2 * r + (q >> 1);
          if (k < 5) ALD("=a", bb[k][e], voffB + (q >> 1) * SLICE_BYTES, ROFF(k), e * 1024);
          else ALD("=v", bb[k][e], voffB + (q >> 1) * SLICE_BYTES, ROFF(k), e * 1024);
        }
    }
    STAMP(7);

    if (has5 && (got & 0xffull) != 0xffull) got = wait_flags(flc, need, 0xffull, tmo, call, bad);
    u4 aa[16];
#pragma unroll
    for (int k = 0; k < 16; ++k) ALD("=a", aa[k], has5 ? vl + sx * 2048 : OOB, ROFF(k), (k & 1) * 1024);
    APOLL(fv, flc);
    asm volatile(FILL_ FILL_ FILL_ FILL_ :: [oob] "v"(OOB), [rq] "s"(rq) : "memory");
    AWAIT(20, "+a", aa[0]);
#pragma unroll
    for (int k = 0; k < 16; ++k) {
      if (k < 5) asm volatile("" : "+a"(bb[k][0]), "+a"(bb[k][1]), "+a"(bb[k][2]), "+a"(bb[k][3]));
      else asm volatile("" : "+v"(bb[k][0]), "+v"(bb[k][1]), "+v"(bb[k][2]), "+v"(bb[k][3]));
    }
#pragma unroll
    for (int i = 0; i < 5; ++i) {
      f4* c = acc[i == 0 ? 4 : i - 1];
      const unsigned long long wantn = 0xff00ull << (8 * i);
      const int voffN = vl + u * 8192 + i * 2048;
      const bool pfa = (got & wantn) == wantn;
      bool pfb = pfa;
      int voffP = pfa ? voffN : OOB;
#pragma unroll
      for (int k = 0; k < 16; ++k) {
        if (i < 4 && k == 8) {
          if (!pfa) {
            AWAIT(12, "+v", fv);
            got |= __ballot(fv >= need);
            pfb = (got & wantn) == wantn;
          }
          const int cond = dry || (pfb && !pfa);
          asm volatile("s_cmp_eq_u32 %[c], 0\n\ts_cbranch_scc1 1f\n\t" LD8_ "s_branch 2f\n1:\n\t"
                       FILL_ FILL_ FILL_ FILL_ FILL_ FILL_ FILL_ FILL_ "\n2:"
                       : LD8_OUT(0) : LD8_IN(0, voffN, cond) : "memory", "scc");
          voffP = pfb ? voffN : OOB;
        }
        AWAIT(i == 4 ? 20 - k : k < 8 ? 20 : 28, "+a", aa[k]);
        __builtin_amdgcn_sched_barrier(0);
#pragma unroll
        for (int e = 0; e < 4; ++e) c[e] = MFMA16(aa[k], bb[k][e], c[e]);
        __builtin_amdgcn_sched_barrier(0);
        if (i < 4) ALD("+a", aa[k], voffP, ROFF(k), (k & 1) * 1024);
      }
      if (i < 4) APOLL(fv, flc); else AWAIT(4, "+v", fv);
      __builtin_amdgcn_sched_barrier(0);
      const int vo = (i == 0 && !has5) || EXPM == 3 ? OOB : voffO + (i == 0 ? sx : 4 * u + i - 1) * 16 * NN * 4;
      const float one = bad ? __builtin_nanf("") : 1.0f;
#pragma unroll
      for (int r = 0; r < 4; ++r) {
        f4 y;
#pragma unroll
        for (int e = 0; e < 4; ++e) y[e] = gelu_tanh(c[e][r], one);
        __builtin_amdgcn_raw_buffer_store_b128(__builtin_bit_cast(u4, y), ro, vo + r * NN * 4, 0, OUTAUX);
      }
      __builtin_amdgcn_sched_barrier(0);
      if (i < 4) {
        const int cond = dry || !pfb;
        if (cond) {
          AWAIT(4, "+v", fv);
          got |= __ballot(fv >= need);
          if ((got & wantn) != wantn) got = wait_flags(flc, need, wantn, tmo, call, bad);
        }
        asm volatile("s_cmp_eq_u32 %[c], 0\n\ts_cbranch_scc1 1f\n\t" LD8_ "\n1:" : LD8_OUT(0) : LD8_IN(0, voffN, cond) : "memory", "scc");
        asm volatile("s_cmp_eq_u32 %[c], 0\n\ts_cbranch_scc1 1f\n\t" LD8_ "global_load_dword %[fv], %[fl], off " POLL_BITS "\n\t" FILL_ FILL_ FILL_ FILL_ "\n1:"
                     : LD8_OUT(8), [fv] "+v"(fv) : LD8_IN(8, voffN, cond), [fl] "v"(flc) : "memory", "scc");
      }
    }
    if (dry) STAMP(15);
  }
  STAMP(14);
}

__global__ void __launch_bounds__(256, 1) gemm_ag_gelu(Params p) {
  const int lane = threadIdx.x & 63;
  const int wave = __builtin_amdgcn_readfirstlane(threadIdx.x >> 6);
  const int b = blockIdx.x;
  STAMP(0);
  if (b < NPWG) {
    if (wave < PW) pusher(p, wave * NPWG + b, lane);
  } else {
    const int xcd = b & 7, l = (xcd >> 1) * NG + (b >> 3) - NPWG / 8;
    const int ti = l >> 3, tj = (xcd & 1) * 8 + (l & 7);
    consumer(p, 2 * ti + (wave >> 1), 2 * tj + (wave & 1), lane);
  }
}

extern "C" void kernel_launch(int rank, int nranks, ncclComm_t comm,
                              hipStream_t stream,
                              void* const* d_in, const int* in_sizes, int n_in,
                              void* d_out, int out_size,
                              const int* in_dtypes, int out_dtype,
                              void* const* d_out_all,
                              void* const* d_ws_all, size_t ws_bytes) {
  if (nranks != NR || (unsigned)rank >= (unsigned)NR || n_in < 2 || ws_bytes < WS_NEED ||
      in_sizes[0] != MM * KS || in_sizes[1] != KS * NN || out_size != MM * NN) {
    fprintf(stderr, "kernel_launch: unexpected problem shape\n");
    return;
  }
  Params p;
  p.x = (const float*)d_in[0];
  p.w = (const float*)d_in[1];
  p.out = (float*)d_out;
  for (int q = 0; q < 2 * NR; ++q) p.ws2[q] = (char*)d_ws_all[q & (NR - 1)] + DATA_OFF;
  p.myws = (char*)d_ws_all[rank];
  p.rank = rank;
  gemm_ag_gelu<<<256, 256, 0, stream>>>(p);
  HIP_CHECK(hipGetLastError());
}
```

```cpp
#include <hip/hip_runtime.h>
#include <rccl/rccl.h>
#include <cstdio>

#define HIP_CHECK(expr) do { hipError_t _e = (expr); if (_e != hipSuccess) { \
    fprintf(stderr, "HIP error %s at %s:%d\n", hipGetErrorString(_e), __FILE__, __LINE__); \
    return; } } while (0)

typedef _Float16 h8 __attribute__((ext_vector_type(8)));
typedef float f2 __attribute__((ext_vector_type(2)));
typedef float f4 __attribute__((ext_vector_type(4)));
typedef unsigned u4 __attribute__((ext_vector_type(4)));
typedef __attribute__((address_space(1))) unsigned gu32;
typedef __attribute__((address_space(1))) unsigned long long gu64;

#define DD 4
#define JS1 16
#define PMASK 31
#define XSLEEP 2
#define WSLEEP 8
#define W7SLEEP 2
#define OUTAUX 16
#define POLL_SCOPE __HIP_MEMORY_SCOPE_SYSTEM
#define POLL_BITS "sc0 sc1"
#define LDBITS "sc1"

constexpr bool in_v(int r, int s, int e) { return s == 7 || (r == 0 && s * 4 + e < 6); }
constexpr int NR = 8;
constexpr int MM = 2048, NN = 2048, KS = 256;
constexpr int NPUSH = 32;
constexpr int TW4 = 96, TW = TW4 + 64;
constexpr int TP = TW + 4 + 14 * (JS1 - 2);
constexpr int NT = TP + 28 * (20 - JS1);
constexpr unsigned FLW_OFF = 0;
constexpr unsigned FLX_OFF = 16384;
constexpr unsigned TKT_OFF = 49152;
constexpr unsigned CLR_OFF = 51200;
constexpr int NCW = 896;
constexpr unsigned CNT_OFF = 53248;
constexpr unsigned DATA_OFF = 1u << 20;
constexpr unsigned SRC_BYTES = 2u << 20;
constexpr unsigned W_PART = 1u << 20;
constexpr unsigned PAR_BYTES = NR * SRC_BYTES;
constexpr size_t WS_NEED = (size_t)DATA_OFF + 2ull * PAR_BYTES;
constexpr unsigned MAX_POLLS = 1u << 22;
constexpr unsigned MAX_SPINS = 1u << 21;
constexpr int AUX_SYS = 17;
constexpr int OOB = 0x70000000;

struct Params {
  const float* x; const float* w; float* out;
  char* myws;
  char* ws2[2 * NR];
  int rank;
};

#define STAMP(i) do {} while (0)

template <int N> __device__ __forceinline__ void wait_vm() { asm volatile("s_waitcnt vmcnt(%0)" ::"i"(N) : "memory"); }

__device__ __forceinline__ __amdgpu_buffer_rsrc_t make_rsrc(const void* p, unsigned bytes) {
  return __builtin_amdgcn_make_buffer_rsrc((void*)p, 0, (int)bytes, 0x00020000);
}

__device__ __forceinline__ gu32* call_counter(char* myws) { return (gu32*)(myws + CNT_OFF + (blockIdx.x * 4 + (threadIdx.x >> 6)) * 4); }

struct Item {
  int isw;
  int src;
  int dst;
  int flag;
};

__device__ __forceinline__ Item make_item(int t, int i, int rank) {
  Item it;
  const int c = t & 31, part = t >> 5, s = t < TW4 ? 2 * part + (i >> 1) : part + 3;
  it.isw = t < TW;
  const bool pair = t < TP;
  const int y = pair ? t - TW + 24 : t - TP + 28 * JS1;
  const int a = y / 7;
  int g = y - 7 * a, js = pair ? a >> 1 : a >> 2, q = pair ? 2 * (a & 1) + i : a & 3;
  if (y < 28) { g = 5 + (y & 1); q = (y & 2) + i; }
  const int strip = 7 * js + g - 12;
  it.src = it.isw ? (32 * s * NN + 64 * c + 32 * (i & 1)) * 4 : (16 * strip * KS + 64 * q) * 4;
  it.dst = it.isw ? W_PART + c * 32768 + s * 4096 + (i & 1) * 2048 : strip * 8192 + q * 2048;
  const int fw = i == (t < TW4 ? 3 : 1) ? (int)FLW_OFF + (c * 4 + (rank >> 1)) * 128 + (rank & 1) * 32 + part * 4 + 4 : -1;
  it.flag = t < 128 ? fw : !pair || i ? 0 : -1;
  return it;
}

__device__ __forceinline__ void load_item(__amdgpu_buffer_rsrc_t rx, __amdgpu_buffer_rsrc_t rw, const Item& it, int voff, f2 (&raw)[8]) {
  const int a = it.isw ? 16 * NN * 4 : 32 * 4, b = it.isw ? 2 * NN * 4 : 16 * 4, c = it.isw ? NN * 4 : 2 * 4;
  const __amdgpu_buffer_rsrc_t r = it.isw ? rw : rx;
#pragma unroll
  for (int j = 0; j < 8; ++j)
    raw[j] = __builtin_bit_cast(f2, __builtin_amdgcn_raw_buffer_load_b64(r, voff, it.src + (j >> 2) * a + ((j >> 1) & 1) * b + (j & 1) * c, 0));
}

__device__ __forceinline__ unsigned pk(float a, float b) {
  typedef _Float16 h2 __attribute__((ext_vector_type(2)));
  h2 r; r[0] = (_Float16)a; r[1] = (_Float16)b;
  return __builtin_bit_cast(unsigned, r);
}

__device__ __forceinline__ void pusher(const Params& p, int w, int lane) {
  const __amdgpu_buffer_rsrc_t rx = make_rsrc(p.x, MM * KS * 4), rw = make_rsrc(p.w, KS * NN * 4);
  const __amdgpu_buffer_rsrc_t rc = make_rsrc(p.myws, DATA_OFF);
  const int kq = lane >> 4, l16 = lane & 15;
  const int vw = (4 * kq * NN + 2 * l16) * 4, vx = (l16 * KS + 4 * kq) * 4;
  f2 raw[8];
  int t = w, i = 0;
  Item nxt = make_item(t, 0, p.rank);
  load_item(rx, rw, nxt, vw, raw);
  gu32* cnt = call_counter(p.myws);
  const unsigned cnt0 = __hip_atomic_load(cnt, __ATOMIC_RELAXED, __HIP_MEMORY_SCOPE_AGENT);
  char* fp = p.ws2[lane & (NR - 1)] - DATA_OFF;
  __builtin_amdgcn_sched_barrier(0);
  __amdgpu_buffer_rsrc_t rs[NR];
  char* const* tab = p.ws2 + (w & (NR - 1));
#pragma unroll
  for (int q = 0; q < NR; ++q) rs[q] = make_rsrc(tab[q], 2 * PAR_BYTES);
  const unsigned call = __builtin_amdgcn_readfirstlane(cnt0) + 1u;
  __hip_atomic_store(cnt, call, __ATOMIC_RELAXED, __HIP_MEMORY_SCOPE_AGENT);
  STAMP(20);
  const int dbase = (call & 1u) * PAR_BYTES + p.rank * SRC_BYTES;
  const unsigned tbase = (call - 1u) * NT - NPUSH;
  const int voff = lane * 16;
  const int vtk = lane == 0 ? (int)TKT_OFF : OOB;
  int tk = 0;
  int pflag = -1;
  u4 gate = {0u, 0u, 0u, 0u};
  bool open = false;
  const int vgt = lane < 56 ? (int)CLR_OFF + lane * 16 : OOB;
  auto publish = [&]() {
    if (lane < NR) __hip_atomic_store((gu32*)(fp + pflag), call, __ATOMIC_RELAXED, __HIP_MEMORY_SCOPE_SYSTEM);
  };

#pragma unroll 1
  for (int j = 0;; ++j) {
    const Item cur = nxt;
    u4 fr[2];
#pragma unroll
    for (int e = 0; e < 2; ++e)
#pragma unroll
      for (int d = 0; d < 4; ++d) {
        const unsigned cw = pk(raw[2 * d][e], raw[2 * d + 1][e]), cx = pk(raw[4 * e + d][0], raw[4 * e + d][1]);
        fr[e][d] = cur.isw ? cw : cx;
      }
    fr[0][0] = fr[0][0] ? fr[0][0] : 0x8000u;
    fr[1][0] = fr[1][0] ? fr[1][0] : 0x8000u;
    if (!cur.isw && !open) {
      for (unsigned n = 0;; ++n) {
        const unsigned lo = min(min(gate[0], gate[1]), min(gate[2], gate[3]));
        if ((__ballot(lo >= call) | 0xffull << 56) == ~0ull) break;
        if (n > MAX_POLLS) return;
        asm volatile("global_load_dwordx4 %0, %1, off sc0 sc1\n\ts_waitcnt vmcnt(0)" : "=v"(gate) : "v"(p.myws + CLR_OFF + (lane < 56 ? lane : 0) * 16) : "memory");
      }
      open = true;
    }
    if (j == 0) STAMP(21);
    bool more = true;
    ++i;
    if (cur.flag >= 0) {
      const unsigned tn = (unsigned)__builtin_amdgcn_readfirstlane(tk) - tbase;
      more = tn < (unsigned)NT;
      t = more ? (int)tn : 0;
      i = 0;
    }
    nxt = make_item(t, i, p.rank);
    __builtin_amdgcn_sched_barrier(0);
    load_item(rx, rw, nxt, !more ? OOB : nxt.isw ? vw : vx, raw);
    tk = __builtin_amdgcn_raw_ptr_buffer_atomic_add_i32(1, rc, (more && nxt.flag >= 0) ? vtk : OOB, 0, 0);
    gate = __builtin_amdgcn_raw_buffer_load_b128(rc, !nxt.isw && !open ? vgt : OOB, 0, AUX_SYS);
    __builtin_amdgcn_sched_barrier(0);
#pragma unroll
    for (int e = 0; e < 2; ++e) {
#pragma unroll
      for (int q = 0; q < NR; ++q)
        __builtin_amdgcn_raw_buffer_store_b128(fr[e], rs[q], voff, dbase + cur.dst + e * 1024, AUX_SYS);
      wait_vm<26>();
      if (e == 1 && pflag >= 0) publish();
    }
    pflag = cur.flag > 0 ? cur.flag - 4 : -1;
    if (!more) break;
  }
  wait_vm<0>();
  if (pflag >= 0) publish();
  STAMP(11);
}

#define MFMA16(a, b, c) __builtin_amdgcn_mfma_f32_16x16x32_f16(__builtin_bit_cast(h8, a), __builtin_bit_cast(h8, b), c, 0, 0, 0)
#define ALD(c, dst, voff, soff, imm) asm volatile("buffer_load_dwordx4 %0, %1, %2, %3 offen offset:%4 " LDBITS : c(dst) : "v"(voff), "s"(rq), "s"(soff), "i"(imm) : "memory")
#define WAIT_FLAGS(ptr, want, slp) \
  while ((got & (want)) != (want)) { \
    got = (unsigned)__ballot(__hip_atomic_load(ptr, __ATOMIC_RELAXED, POLL_SCOPE) >= need) & (unsigned)((2ull << PMASK) - 1); \
    if ((got & (want)) == (want)) break; \
    __builtin_amdgcn_s_sleep(slp); \
    if (++polls > MAX_POLLS) need = 0u; \
  }

template <int I> struct IC { static constexpr int value = I; };
template <int I, int N, class F> __device__ __forceinline__ void sfor(F&& f) { if constexpr (I < N) { f(IC<I>{}); sfor<I + 1, N>(f); } }
#define INL __attribute__((always_inline))
#define PIN(x) asm volatile("" : "+v"(x))
#define ADD(d, a, b) asm volatile("v_add_f32 %0, %1, %2" : "=v"(d) : "v"(a), "v"(b))
#define ADDS(d, a, b) asm volatile("v_add_f32 %0, %1, %2" : "=v"(d) : "s"(a), "v"(b))
#define MUL(d, a, b) asm volatile("v_mul_f32 %0, %1, %2" : "=v"(d) : "v"(a), "v"(b))
#define FMA(d, a, b, c) asm volatile("v_fma_f32 %0, %1, %2, %3" : "=v"(d) : "v"(a), "s"(b), "v"(c))
constexpr int LPART = 4864, LBUF = 4 * LPART;

#define PINS(x) asm volatile("" : "+s"(x))
#define RARE(x) __builtin_expect(x, 0)

__device__ __forceinline__ void consumer(const Params& p, int g, int c, int v, int lane, char* lds) {
  static_assert(DD >= 3 && DD <= 5, "");
  char* myws = p.myws;
  gu32* cnt = call_counter(myws);
  unsigned need = __builtin_amdgcn_readfirstlane(__hip_atomic_load(cnt, __ATOMIC_RELAXED, __HIP_MEMORY_SCOPE_AGENT)) + 1u;
  __hip_atomic_store(cnt, need, __ATOMIC_RELAXED, __HIP_MEMORY_SCOPE_AGENT);
  const unsigned long long rsb = (unsigned long long)(myws + DATA_OFF + (need & 1u) * PAR_BYTES + 2 * v * SRC_BYTES);
  const u4 rq = {(unsigned)rsb, (unsigned)(rsb >> 32) & 0xffffu, 2 * SRC_BYTES, 0x00020000u};
  const gu32* flw = (const gu32*)(myws + FLW_OFF) + (c * 4 + v) * 32 + (lane & 15);
  const int voffX = lane * 16 + (g - 12) * 8192, voffW = lane * 16 + W_PART + c * 32768;
  const int voffO = ((4 * v + (lane >> 4)) * NN + 64 * c + 4 * (lane & 15)) * 4, soO = 16 * (g - 12) * NN * 4;
  const int jmin = g >= 5 ? 1 : 2;
  unsigned polls = 0, got, left = MAX_SPINS, one = 0x3f800000u, live = 1;
  int so[4] = {0, 4096, (int)SRC_BYTES, (int)SRC_BYTES + 4096};
  PINS(so[0]); PINS(so[1]); PINS(so[2]); PINS(so[3]);
  u4 bb[2][8][4], aa[8][4];
#pragma unroll
  for (int k = 0; k < 64; ++k) {
    if (in_v(k >> 5, (k >> 2) & 7, k & 3)) asm volatile("" : "=v"(bb[k >> 5][(k >> 2) & 7][k & 3]));
    else asm volatile("" : "=a"(bb[k >> 5][(k >> 2) & 7][k & 3]));
  }
#pragma unroll
  for (int k = 0; k < 32; ++k) asm volatile("" : "=v"(aa[k >> 2][k & 3]));
  f4 acc[2][4], s0, s1, s2, s3;
#pragma unroll
  for (int k = 0; k < 4; ++k) acc[1][k] = (f4){0.f, 0.f, 0.f, 0.f};
  int voffN[2] = {OOB, OOB}, voffD = OOB;
  unsigned thr[2] = {0u, 0u}, szO, soS, tmin;
  asm volatile("" : "=v"(tmin));
#define KEEP_AA _Pragma("unroll") for (int k = 0; k < DD; ++k) asm volatile("" :: "v"(aa[k][0]), "v"(aa[k][1]), "v"(aa[k][2]), "v"(aa[k][3]) : "memory")
  const float k0 = -2.0f * 0.7978845608028654f * 1.4426950408889634f, k1 = k0 * 0.044715f;
  char* lw = lds + v * LPART + (lane >> 4) * 288 + (lane & 15) * 16;
  const char* lr = lds + ((lane >> 3) & 1) * 2432 + ((lane >> 1) & 3) * 288 + (lane & 1) * 8 + (4 * v + (lane >> 4)) * 16;
  auto piece = [&](auto gp_, auto pb_, int js) INL {
    constexpr int gp = gp_.value, pb = pb_.value, m = gp & 15;
    constexpr int wf = gp == 0 ? 0 : gp == 2 ? 1 : gp == 3 ? 2 : gp == 5 ? 3 : -1;
    if constexpr (wf >= 0) *(f4*)(lw + pb * LBUF + wf * 1152 + (wf >> 1) * 128) = acc[pb][wf];
    if constexpr (gp == 9) asm volatile("s_waitcnt lgkmcnt(0)\n\ts_barrier" ::: "memory");
    auto rd = [&](int part) INL {
      const f2 a = *(const f2*)(lr + pb * LBUF + part * LPART), b = *(const f2*)(lr + pb * LBUF + part * LPART + 1152);
      return (f4){a[0], a[1], b[0], b[1]};
    };
    if constexpr (gp == 11) { s0 = rd(0); s1 = rd(1); }
    if constexpr (gp == 12) { s2 = rd(2); s3 = rd(3); }
    if constexpr (gp == 16) { szO = js >= jmin ? MM * NN * 4 : 0; soS = soO + js * (7 * 16 * NN * 4); PINS(szO); PINS(soS); }
    constexpr int z = gp == 32 ? 8 : gp < 16 || m == 0 || m > 12 || m % 3 == 1 ? -1 : (gp >> 4) * 8 - 8 + (gp > 32) + m - 2 - (m - 2) / 3, e = 2 * (z & 1);
    if constexpr (z == 0 || z == 1) { ADD(s0[e], s0[e], s1[e]); ADD(s0[e + 1], s0[e + 1], s1[e + 1]); }
    if constexpr (z == 2 || z == 3) { ADD(s2[e], s2[e], s3[e]); ADD(s2[e + 1], s2[e + 1], s3[e + 1]); }
    if constexpr (z == 4 || z == 5) { ADD(s0[e], s0[e], s2[e]); ADD(s0[e + 1], s0[e + 1], s2[e + 1]); }
    if constexpr (z == 6 || z == 7) { MUL(s1[e], s0[e], s0[e]); MUL(s1[e + 1], s0[e + 1], s0[e + 1]); }
    if constexpr (z == 8 || z == 9) { FMA(s1[e], s1[e], k1, k0); FMA(s1[e + 1], s1[e + 1], k1, k0); }
    if constexpr (z == 10 || z == 11) { MUL(s1[e], s1[e], s0[e]); MUL(s1[e + 1], s1[e + 1], s0[e + 1]); }
    if constexpr (z >= 12 && z < 16) { s1[z - 12] = __builtin_amdgcn_exp2f(s1[z - 12]); PIN(s1); }
    if constexpr (z == 16 || z == 17) { ADDS(s1[e], one, s1[e]); ADDS(s1[e + 1], one, s1[e + 1]); }
    if constexpr (z >= 18 && z < 22) { s1[z - 18] = __builtin_amdgcn_rcpf(s1[z - 18]); PIN(s1); }
    if constexpr (z == 22 || z == 23) { MUL(s2[e >> 1], s1[e], s0[e]); MUL(s2[(e >> 1) + 2], s1[e + 1], s0[e + 1]); }
    if constexpr (z == 24) {
      __builtin_amdgcn_raw_buffer_store_b128(__builtin_bit_cast(u4, s2), make_rsrc(p.out, szO), voffO, soS, OUTAUX);
    }
  };
#define LD_(d, s, o) "buffer_load_dwordx4 %[d" #d "], %[vo], %[rq], %[" #s "] offen offset:%[" #o "] " LDBITS "\n\t"
#define FIRSTS(n) [a0] "v"(aa[(n) & 7][0][0]), [a1] "v"(aa[(n) & 7][1][0]), [a2] "v"(aa[(n) & 7][2][0]), [a3] "v"(aa[(n) & 7][3][0])
  auto again = [&](auto u_, auto id_) INL {
    constexpr int u = u_.value, id = id_.value;
    (void)aa; (void)rq; (void)so; (void)voffN; (void)left; (void)thr; (void)live; (void)one; (void)tmin;
    asm volatile(".subsection 1\n.Lslow%c0:\n\ts_sleep %1\n\t.subsection 0" :: "i"(id), "i"(XSLEEP));
    sfor<u, u + DD>([&](auto n_) INL {
      constexpr int n = n_.value, qn = n & 3;
      (void)aa; (void)rq; (void)so; (void)voffN;
      asm volatile(".subsection 1\n\t" LD_(0, sa, oa) LD_(1, sa, ob) LD_(2, sb, oa) LD_(3, sb, ob) ".subsection 0"
                   :: [d0] "v"(aa[n & 7][0]), [d1] "v"(aa[n & 7][1]), [d2] "v"(aa[n & 7][2]), [d3] "v"(aa[n & 7][3]),
                      [vo] "v"(voffN[(n >> 2) & 1]), [rq] "s"(rq), [sa] "s"(so[qn >> 1]), [sb] "s"(so[2 + (qn >> 1)]),
                      [oa] "i"((qn & 1) * 2048), [ob] "i"((qn & 1) * 2048 + 1024) : "memory");
    });
    asm volatile(".subsection 1\n\ts_sub_u32 %[left], %[left], 1\n\ts_cbranch_scc0 .Lgo%c[id]\n\t"
                 "s_mov_b32 %[t0], 0\n\ts_mov_b32 %[t1], 0\n\ts_mov_b32 %[live], 0\n\ts_mov_b32 %[one], 0x7fc00000\n.Lgo%c[id]:\n\t"
                 "s_waitcnt vmcnt(%[n])\n\tv_min3_u32 %[t], %[a0], %[a1], %[a2]\n\tv_min_u32 %[t], %[t], %[a3]\n\t"
                 "v_cmp_gt_u32 vcc, %[t0], %[t]\n\ts_cbranch_vccnz .Lslow%c[id]\n\ts_branch .Lback%c[id]\n\t.subsection 0\n\t"
                 "s_cbranch_vccnz .Lslow%c[id]\n.Lback%c[id]:"
                 : [left] "+s"(left), [t0] "+s"(thr[(u >> 2) & 1]), [t1] "+s"(thr[((u >> 2) & 1) ^ 1]), [live] "+s"(live), [one] "+s"(one)
                 : FIRSTS(u), [t] "v"(tmin), [n] "i"(4 * DD - 4), [id] "i"(id) : "memory", "scc", "vcc");
  };

  PINS(thr[0]); PINS(thr[1]);
  int j0 = -4, jend = -2;
#pragma unroll 1
  for (;;) {
    sfor<0, 8>([&](auto st_) INL {
      constexpr int st = st_.value, q = st & 3, pb = st >> 2, n = st + DD, u = st + 1;
      constexpr int qn = n & 3, par = (n >> 2) & 1;
      sfor<0, 16>([&](auto m_) INL {
        constexpr int m = m_.value, k = m >> 2, e = m & 3;
        __builtin_amdgcn_sched_barrier(0);
        acc[pb][e] = MFMA16(bb[k >> 1][2 * q + (k & 1)][e], aa[st][k], (q == 0 && k == 0 ? (f4){0.f, 0.f, 0.f, 0.f} : acc[pb][e]));
        __builtin_amdgcn_sched_barrier(0);
        if constexpr (q == 3 && m == 0) { voffN[pb] = voffD + (j0 + pb + 2) * (7 * 8192); PIN(voffN[pb]); }
        if constexpr (m % 3 == 1 && m <= 10) {
          constexpr int kk = m / 3;
          asm volatile("buffer_load_dwordx4 %0, %1, %2, %3 offen offset:%4 " LDBITS : "=v"(aa[n & 7][kk])
                       : "v"(voffN[par]), "s"(rq), "s"(so[(kk & 2) + (qn >> 1)]), "i"((qn & 1) * 2048 + (kk & 1) * 1024) : "memory");
        }
        if constexpr (q == 0 && m == 6) { thr[pb ^ 1] = (unsigned)(j0 + pb + 1) < 20u ? live : 0u; PINS(thr[pb ^ 1]); }
        if constexpr (m == 13)
          asm volatile("s_waitcnt vmcnt(%[n])\n\tv_min3_u32 %[t], %[a0], %[a1], %[a2]" :: FIRSTS(u), [t] "v"(tmin), [n] "i"(4 * DD - 4) : "memory");
        if constexpr (m == 14)
          asm volatile("v_min_u32 %[t], %[t], %[a3]\n\tv_cmp_gt_u32 vcc, %[t0], %[t]" :: FIRSTS(u), [t] "v"(tmin), [t0] "s"(thr[(u >> 2) & 1]) : "memory", "vcc");
        if constexpr (m == 15) {
          again(IC<u>{}, IC<st>{});
          asm volatile("" : "+v"(aa[u & 7][0]), "+v"(aa[u & 7][1]), "+v"(aa[u & 7][2]), "+v"(aa[u & 7][3]));
        }
        piece(IC<16 * q + m>{}, IC<(pb ^ 1)>{}, j0 + pb - 1);
        __builtin_amdgcn_sched_barrier(0);
      });
    });
    j0 += 2;
    if (RARE(j0 == jend)) {
      sfor<0, 64>([&](auto i_) INL { piece(i_, IC<1>{}, j0 - 1); });
      asm volatile("s_cmp_lt_i32 %0, 0\n\ts_cbranch_scc1 1f\n\ts_endpgm\n1:" :: "s"(j0) : "memory", "scc");
      {
        {
          const __amdgpu_buffer_rsrc_t rz = make_rsrc(myws + DATA_OFF + ((need ^ 1u) & 1u) * PAR_BYTES, PAR_BYTES);
#pragma unroll
          for (int k = 0; k < 11; ++k) {
            const int f = (g * 32 + c) * 4 + v + k * NCW, fw = f - 8192;
            __builtin_amdgcn_raw_buffer_store_b128((u4){0u, 0u, 0u, 0u}, rz, fw < 1024 ? lane * 16 : OOB,
              fw < 0 ? (f >> 10) * SRC_BYTES + (f & 1023) * 1024 : ((fw >> 7) & 7) * SRC_BYTES + W_PART + ((fw >> 2) & 31) * 32768 + 7 * 4096 + (fw & 3) * 1024, AUX_SYS);
          }
        }
        wait_vm<0>();
        KEEP_AA;
        __builtin_amdgcn_s_barrier();
        __builtin_amdgcn_raw_buffer_store_b32(need, make_rsrc(myws, DATA_OFF), lane == 0 && v == 0 ? (int)CLR_OFF : OOB, (g * 32 + c) * 4, AUX_SYS);
        STAMP(15);
        got = 0;
#pragma unroll
        for (int q = 0; q < 4; ++q) {
          if (q < 3) { WAIT_FLAGS(flw, 0x101u << q, WSLEEP); } else { WAIT_FLAGS(flw, 0x101u << q, 1); }
          asm volatile("" ::: "memory");
          STAMP(1 + q);
#pragma unroll
          for (int k = 0; k < (q < 3 ? 16 : 8); ++k) {
            const int s = q < 3 ? 2 * q + (k >> 3) : q + 3, r = (k >> 2) & 1;
            if (in_v(r, s, k & 3)) ALD("=v", bb[r][s][k & 3], voffW, r * SRC_BYTES + s * 4096, (k & 3) * 1024);
            else ALD("=a", bb[r][s][k & 3], voffW, r * SRC_BYTES + s * 4096, (k & 3) * 1024);
          }
        }
        for (;;) {
#pragma unroll
          for (int k = 0; k < 8; ++k) ALD("=v", bb[k >> 2][7][k & 3], voffW, (k >> 2) * SRC_BYTES + 7 * 4096, (k & 3) * 1024);
          asm volatile("s_waitcnt vmcnt(0)" : "+v"(bb[0][7][0]), "+v"(bb[0][7][1]), "+v"(bb[0][7][2]), "+v"(bb[0][7][3]),
                                              "+v"(bb[1][7][0]), "+v"(bb[1][7][1]), "+v"(bb[1][7][2]), "+v"(bb[1][7][3]) :: "memory");
          unsigned lo = bb[0][7][0][0];
#pragma unroll
          for (int k = 1; k < 8; ++k) lo = min(lo, bb[k >> 2][7][k & 3][0]);
          if (__ballot(lo == 0u) == 0ull || need == 0u) break;
          __builtin_amdgcn_s_sleep(W7SLEEP);
          if (++polls > MAX_POLLS) need = 0u;
        }
        STAMP(7);
        one = need ? one : 0x7fc00000u;
        asm volatile("s_min_u32 %0, %1, 1" : "=s"(live) : "s"(need) : "scc");
        j0 = jmin & 2;
        voffD = voffX;
        voffN[0] = jmin & 1 ? OOB : voffX + j0 * 7 * 8192;
        voffN[1] = voffX + (j0 + 1) * 7 * 8192;
        thr[0] = (jmin >> 1) & live;
        PINS(live); PINS(one); PINS(thr[0]); PIN(voffN[0]); PIN(voffN[1]);
        asm volatile("s_mov_b64 vcc, -1" ::: "vcc");
        again(IC<0>{}, IC<8>{});
        jend = 20;
        PINS(jend);
      }
    }
    asm volatile("" ::: "memory");
  }
}

__global__ void __launch_bounds__(256, 1) gemm_ag_gelu(Params p) {
  __shared__ __attribute__((aligned(16))) char lds[2 * LBUF];
  const int lane = threadIdx.x & 63;
  const int wave = __builtin_amdgcn_readfirstlane(threadIdx.x >> 6);
  const int b = blockIdx.x;
  STAMP(0);
  if (b < NPUSH) {
    if (wave == 0) pusher(p, b, lane);
  } else {
    const int i = (b & 7) * 28 + (b >> 3) - NPUSH / 8;
    consumer(p, i >> 5, i & 31, wave, lane, lds);
  }
}

extern "C" void kernel_launch(int rank, int nranks, ncclComm_t comm,
                              hipStream_t stream,
                              void* const* d_in, const int* in_sizes, int n_in,
                              void* d_out, int out_size,
                              const int* in_dtypes, int out_dtype,
                              void* const* d_out_all,
                              void* const* d_ws_all, size_t ws_bytes) {
  if (nranks != NR || (unsigned)rank >= (unsigned)NR || n_in < 2 || ws_bytes < WS_NEED ||
      in_sizes[0] != MM * KS || in_sizes[1] != KS * NN || out_size != MM * NN) {
    fprintf(stderr, "kernel_launch: unexpected problem shape\n");
    return;
  }
  Params p;
  p.x = (const float*)d_in[0];
  p.w = (const float*)d_in[1];
  p.out = (float*)d_out;
  for (int q = 0; q < 2 * NR; ++q) p.ws2[q] = (char*)d_ws_all[q & (NR - 1)] + DATA_OFF;
  p.myws = (char*)d_ws_all[rank];
  p.rank = rank;
  gemm_ag_gelu<<<256, 256, 0, stream>>>(p);
  HIP_CHECK(hipGetLastError());
}
```

```cpp
#include <hip/hip_runtime.h>
#include <rccl/rccl.h>
#include <cstdio>

#define HIP_CHECK(expr) do { hipError_t _e = (expr); if (_e != hipSuccess) { \
    fprintf(stderr, "HIP error %s at %s:%d\n", hipGetErrorString(_e), __FILE__, __LINE__); \
    return; } } while (0)

typedef _Float16 h8 __attribute__((ext_vector_type(8)));
typedef float f2 __attribute__((ext_vector_type(2)));
typedef float f4 __attribute__((ext_vector_type(4)));
typedef unsigned u4 __attribute__((ext_vector_type(4)));
typedef __attribute__((address_space(1))) unsigned gu32;
typedef __attribute__((address_space(1))) unsigned long long gu64;

#define DD 4
#define JS1 16
#define PMASK 31
#define XSLEEP 2
#define WSLEEP 8
#define W7SLEEP 2
#define OUTAUX 16
#define POLL_SCOPE __HIP_MEMORY_SCOPE_SYSTEM
#define POLL_BITS "sc0 sc1"
#define LDBITS "sc1"

constexpr bool in_v(int r, int s, int e) { return s == 7 || (r == 0 && s * 4 + e < 6); }
constexpr int NR = 8;
constexpr int MM = 2048, NN = 2048, KS = 256;
constexpr int NPUSH = 32;
constexpr int TW4 = 96, TW = TW4 + 64;
constexpr int TP = TW + 4 + 14 * (JS1 - 2);
constexpr int NT = TP + 28 * (20 - JS1);
constexpr unsigned FLW_OFF = 0;
constexpr unsigned FLX_OFF = 16384;
constexpr unsigned TKT_OFF = 49152;
constexpr unsigned CLR_OFF = 51200;
constexpr int NCW = 896;
constexpr unsigned CNT_OFF = 53248;
constexpr unsigned DATA_OFF = 1u << 20;
constexpr unsigned SRC_BYTES = 2u << 20;
constexpr unsigned W_PART = 1u << 20;
constexpr unsigned PAR_BYTES = NR * SRC_BYTES;
constexpr size_t WS_NEED = (size_t)DATA_OFF + 2ull * PAR_BYTES;
constexpr unsigned MAX_POLLS = 1u << 22;
constexpr unsigned MAX_SPINS = 1u << 21;
constexpr int AUX_SYS = 17;
constexpr int OOB = 0x70000000;

struct Params {
  const float* x; const float* w; float* out;
  char* myws;
  char* ws2[2 * NR];
  int rank;
};

#define STAMP(i) do {} while (0)

template <int N> __device__ __forceinline__ void wait_vm() { asm volatile("s_waitcnt vmcnt(%0)" ::"i"(N) : "memory"); }

__device__ __forceinline__ __amdgpu_buffer_rsrc_t make_rsrc(const void* p, unsigned bytes) {
  return __builtin_amdgcn_make_buffer_rsrc((void*)p, 0, (int)bytes, 0x00020000);
}

__device__ __forceinline__ gu32* call_counter(char* myws) { return (gu32*)(myws + CNT_OFF + (blockIdx.x * 4 + (threadIdx.x >> 6)) * 4); }

struct Item {
  int isw;
  int src;
  int dst;
  int flag;
};

__device__ __forceinline__ Item make_item(int t, int i, int rank) {
  Item it;
  const int c = t & 31, part = t >> 5, s = t < TW4 ? 2 * part + (i >> 1) : part + 3;
  it.isw = t < TW;
  const bool pair = t < TP;
  const int y = pair ? t - TW + 24 : t - TP + 28 * JS1;
  const int a = y / 7;
  int g = y - 7 * a, js = pair ? a >> 1 : a >> 2, q = pair ? 2 * (a & 1) + i : a & 3;
  if (y < 28) { g = 5 + (y & 1); q = (y & 2) + i; }
  const int strip = 7 * js + g - 12;
  it.src = it.isw ? (32 * s * NN + 64 * c + 32 * (i & 1)) * 4 : (16 * strip * KS + 64 * q) * 4;
  it.dst = it.isw ? W_PART + c * 32768 + s * 4096 + (i & 1) * 2048 : strip * 8192 + q * 2048;
  const int fw = i == (t < TW4 ? 3 : 1) ? (int)FLW_OFF + (c * 4 + (rank >> 1)) * 128 + (rank & 1) * 32 + part * 4 + 4 : -1;
  it.flag = t < 128 ? fw : !pair || i ? 0 : -1;
  return it;
}

__device__ __forceinline__ void load_item(__amdgpu_buffer_rsrc_t rx, __amdgpu_buffer_rsrc_t rw, const Item& it, int voff, f2 (&raw)[8]) {
  const int a = it.isw ? 16 * NN * 4 : 32 * 4, b = it.isw ? 2 * NN * 4 : 16 * 4, c = it.isw ? NN * 4 : 2 * 4;
  const __amdgpu_buffer_rsrc_t r = it.isw ? rw : rx;
#pragma unroll
  for (int j = 0; j < 8; ++j)
    raw[j] = __builtin_bit_cast(f2, __builtin_amdgcn_raw_buffer_load_b64(r, voff, it.src + (j >> 2) * a + ((j >> 1) & 1) * b + (j & 1) * c, 0));
}

__device__ __forceinline__ unsigned pk(float a, float b) {
  typedef _Float16 h2 __attribute__((ext_vector_type(2)));
  h2 r; r[0] = (_Float16)a; r[1] = (_Float16)b;
  return __builtin_bit_cast(unsigned, r);
}

__device__ __forceinline__ void pusher(const Params& p, int w, int wv, int lane) {
  const __amdgpu_buffer_rsrc_t rx = make_rsrc(p.x, MM * KS * 4), rw = make_rsrc(p.w, KS * NN * 4);
  const __amdgpu_buffer_rsrc_t rc = make_rsrc(p.myws, DATA_OFF);
  const int kq = lane >> 4, l16 = lane & 15;
  const int vw = (4 * kq * NN + 2 * l16) * 4, vx = (l16 * KS + 4 * kq) * 4;
  f2 raw[8], raw2[8];
  const int s0 = wv ? wv - 1 : 2;
  int t = NPUSH + w, i = 0;
  Item nxt = {1, (32 * s0 * NN + 64 * w) * 4, (int)W_PART + w * 32768 + s0 * 4096, -1};
  load_item(rx, rw, nxt, vw, raw);
  load_item(rx, rw, nxt, wv ? vw + 128 : OOB, raw2);
  const int vcnt = CNT_OFF + w * 16 + wv * 4;
  const unsigned cnt0 = __builtin_amdgcn_raw_buffer_load_b32(rc, vcnt, 0, 16);
  char* fp = p.ws2[lane & (NR - 1)];
  __builtin_amdgcn_sched_barrier(0);
  __amdgpu_buffer_rsrc_t rs[NR];
  char* const* tab = p.ws2 + (w & (NR - 1));
#pragma unroll
  for (int q = 0; q < NR; ++q) rs[q] = make_rsrc(tab[q], 2 * PAR_BYTES);
  const unsigned call = __builtin_amdgcn_readfirstlane(cnt0) + 1u;
  const int dbase = (call & 1u) * PAR_BYTES + p.rank * SRC_BYTES;
  const int voff = lane * 16;
  if (wv) {
#pragma unroll
    for (int e = 0; e < 4; ++e) {
      u4 fr;
#pragma unroll
      for (int d = 0; d < 4; ++d) fr[d] = e < 2 ? pk(raw[2 * d][e & 1], raw[2 * d + 1][e & 1]) : pk(raw2[2 * d][e & 1], raw2[2 * d + 1][e & 1]);
#pragma unroll
      for (int q = 0; q < NR; ++q) __builtin_amdgcn_raw_buffer_store_b128(fr, rs[q], voff, dbase + nxt.dst + e * 1024, AUX_SYS);
    }
    __builtin_amdgcn_raw_buffer_store_b32(call, rc, vcnt, 0, 16);
    wait_vm<0>();
    if (lane < NR) __hip_atomic_store((gu32*)(fp + ((int)FLW_OFF + (w * 4 + (p.rank >> 1)) * 128 + (p.rank & 1) * 32 + s0 * 16 - (int)DATA_OFF)), call,
                                      __ATOMIC_RELAXED, __HIP_MEMORY_SCOPE_SYSTEM);
    return;
  }
  __builtin_amdgcn_raw_buffer_store_b32(call, rc, vcnt, 0, 16);
  STAMP(20);
  const unsigned tbase = (call - 1u) * (NT - NPUSH) - 2 * NPUSH;
  const int vtk = lane == 0 ? (int)TKT_OFF : OOB;
  int tk = 0;
  int pflag = -1;
  u4 gate = {0u, 0u, 0u, 0u};
  bool open = false;
  const int vgt = lane < 56 ? (int)CLR_OFF + lane * 16 : OOB;
  auto publish = [&]() {
    if (lane < NR) __hip_atomic_store((gu32*)(fp + (pflag - (int)DATA_OFF)), call, __ATOMIC_RELAXED, __HIP_MEMORY_SCOPE_SYSTEM);
  };

#pragma unroll 1
  for (int j = 0;; ++j) {
    const Item cur = nxt;
    u4 fr[2];
#pragma unroll
    for (int e = 0; e < 2; ++e)
#pragma unroll
      for (int d = 0; d < 4; ++d) {
        const unsigned cw = pk(raw[2 * d][e], raw[2 * d + 1][e]), cx = pk(raw[4 * e + d][0], raw[4 * e + d][1]);
        fr[e][d] = cur.isw ? cw : cx;
      }
    fr[0][0] = fr[0][0] ? fr[0][0] : 0x8000u;
    fr[1][0] = fr[1][0] ? fr[1][0] : 0x8000u;
    if (!cur.isw && !open) {
      for (unsigned n = 0;; ++n) {
        const unsigned lo = min(min(gate[0], gate[1]), min(gate[2], gate[3]));
        if ((__ballot(lo >= call) | 0xffull << 56) == ~0ull) break;
        if (n > MAX_POLLS) return;
        asm volatile("global_load_dwordx4 %0, %1, off sc0 sc1\n\ts_waitcnt vmcnt(0)" : "=v"(gate) : "v"(p.myws + CLR_OFF + (lane < 56 ? lane : 0) * 16) : "memory");
      }
      open = true;
    }
    if (j == 0) STAMP(21);
    bool more = true;
    ++i;
    if (cur.flag >= 0) {
      const unsigned tn = (unsigned)__builtin_amdgcn_readfirstlane(tk) - tbase;
      more = tn < (unsigned)NT;
      t = more ? (int)tn : 0;
      i = 0;
    }
    nxt = make_item(t, i, p.rank);
    __builtin_amdgcn_sched_barrier(0);
    load_item(rx, rw, nxt, !more ? OOB : nxt.isw ? vw : vx, raw);
    tk = __builtin_amdgcn_raw_ptr_buffer_atomic_add_i32(1, rc, (more && nxt.flag >= 0) ? vtk : OOB, 0, 0);
    gate = __builtin_amdgcn_raw_buffer_load_b128(rc, !nxt.isw && !open ? vgt : OOB, 0, AUX_SYS);
    __builtin_amdgcn_sched_barrier(0);
#pragma unroll
    for (int e = 0; e < 2; ++e) {
#pragma unroll
      for (int q = 0; q < NR; ++q)
        __builtin_amdgcn_raw_buffer_store_b128(fr[e], rs[q], voff, dbase + cur.dst + e * 1024, AUX_SYS);
      wait_vm<26>();
      if (e == 1 && pflag >= 0) publish();
    }
    pflag = cur.flag > 0 ? cur.flag - 4 : -1;
    if (!more) break;
  }
  wait_vm<0>();
  if (pflag >= 0) publish();
  STAMP(11);
}

#define MFMA16(a, b, c) __builtin_amdgcn_mfma_f32_16x16x32_f16(__builtin_bit_cast(h8, a), __builtin_bit_cast(h8, b), c, 0, 0, 0)
#define ALD(c, dst, voff, soff, imm) asm volatile("buffer_load_dwordx4 %0, %1, %2, %3 offen offset:%4 " LDBITS : c(dst) : "v"(voff), "s"(rq), "s"(soff), "i"(imm) : "memory")
#define WAIT_FLAGS(ptr, want, slp) \
  while ((got & (want)) != (want)) { \
    got = (unsigned)__ballot(__hip_atomic_load(ptr, __ATOMIC_RELAXED, POLL_SCOPE) >= need) & (unsigned)((2ull << PMASK) - 1); \
    if ((got & (want)) == (want)) break; \
    __builtin_amdgcn_s_sleep(slp); \
    if (++polls > MAX_POLLS) need = 0u; \
  }

template <int I> struct IC { static constexpr int value = I; };
template <int I, int N, class F> __device__ __forceinline__ void sfor(F&& f) { if constexpr (I < N) { f(IC<I>{}); sfor<I + 1, N>(f); } }
#define INL __attribute__((always_inline))
#define PIN(x) asm volatile("" : "+v"(x))
#define ADD(d, a, b) asm volatile("v_add_f32 %0, %1, %2" : "=v"(d) : "v"(a), "v"(b))
#define ADDS(d, a, b) asm volatile("v_add_f32 %0, %1, %2" : "=v"(d) : "s"(a), "v"(b))
#define MUL(d, a, b) asm volatile("v_mul_f32 %0, %1, %2" : "=v"(d) : "v"(a), "v"(b))
#define FMA(d, a, b, c) asm volatile("v_fma_f32 %0, %1, %2, %3" : "=v"(d) : "v"(a), "s"(b), "v"(c))
constexpr int LPART = 4864, LBUF = 4 * LPART;

#define PINS(x) asm volatile("" : "+s"(x))
#define RARE(x) __builtin_expect(x, 0)

__device__ __forceinline__ void consumer(const Params& p, int g, int c, int v, int lane, char* lds) {
  static_assert(DD >= 3 && DD <= 5, "");
  char* myws = p.myws;
  u4 junk;
  asm volatile("" : "=v"(junk));
  gu32* cnt = call_counter(myws);
  unsigned need = __builtin_amdgcn_readfirstlane(__hip_atomic_load(cnt, __ATOMIC_RELAXED, __HIP_MEMORY_SCOPE_AGENT)) + 1u;
  __hip_atomic_store(cnt, need, __ATOMIC_RELAXED, __HIP_MEMORY_SCOPE_AGENT);
#pragma unroll
  for (int k = 0; k < 4; ++k) {
    const int f = 512 + (g * 32 + c) * 4 + v + k * NCW;
    asm volatile("global_load_dwordx4 %0, %1, off" :: "v"(junk), "v"((f < 2048 ? p.w : p.x) + (f & 2047) * 256 + lane * 4), "s"(need) : "memory");
  }
  const unsigned long long rsb = (unsigned long long)(myws + DATA_OFF + (need & 1u) * PAR_BYTES + 2 * v * SRC_BYTES);
  const u4 rq = {(unsigned)rsb, (unsigned)(rsb >> 32) & 0xffffu, 2 * SRC_BYTES, 0x00020000u};
  const gu32* flw = (const gu32*)(myws + FLW_OFF) + (c * 4 + v) * 32 + (lane & 15);
  const int voffX = lane * 16 + (g - 12) * 8192, voffW = lane * 16 + W_PART + c * 32768;
  const int voffO = ((4 * v + (lane >> 4)) * NN + 64 * c + 4 * (lane & 15)) * 4, soO = 16 * (g - 12) * NN * 4;
  const int jmin = g >= 5 ? 1 : 2;
  unsigned polls = 0, got, left = MAX_SPINS, one = 0x3f800000u, live = 1;
  int so[4] = {0, 4096, (int)SRC_BYTES, (int)SRC_BYTES + 4096};
  PINS(so[0]); PINS(so[1]); PINS(so[2]); PINS(so[3]);
  u4 bb[2][8][4], aa[8][4];
#pragma unroll
  for (int k = 0; k < 64; ++k) {
    if (in_v(k >> 5, (k >> 2) & 7, k & 3)) asm volatile("" : "=v"(bb[k >> 5][(k >> 2) & 7][k & 3]));
    else asm volatile("" : "=a"(bb[k >> 5][(k >> 2) & 7][k & 3]));
  }
#pragma unroll
  for (int k = 0; k < 32; ++k) asm volatile("" : "=v"(aa[k >> 2][k & 3]));
  f4 acc[2][4], s0, s1, s2, s3;
#pragma unroll
  for (int k = 0; k < 4; ++k) acc[1][k] = (f4){0.f, 0.f, 0.f, 0.f};
  int voffN[2] = {OOB, OOB}, voffD = OOB;
  unsigned thr[2] = {0u, 0u}, szO, soS, tmin;
  asm volatile("" : "=v"(tmin));
#define KEEP_AA _Pragma("unroll") for (int k = 0; k < DD; ++k) asm volatile("" :: "v"(aa[k][0]), "v"(aa[k][1]), "v"(aa[k][2]), "v"(aa[k][3]) : "memory")
  const float k0 = -2.0f * 0.7978845608028654f * 1.4426950408889634f, k1 = k0 * 0.044715f;
  char* lw = lds + v * LPART + (lane >> 4) * 288 + (lane & 15) * 16;
  const char* lr = lds + ((lane >> 3) & 1) * 2432 + ((lane >> 1) & 3) * 288 + (lane & 1) * 8 + (4 * v + (lane >> 4)) * 16;
  auto piece = [&](auto gp_, auto pb_, int js) INL {
    constexpr int gp = gp_.value, pb = pb_.value, m = gp & 15;
    constexpr int wf = gp == 0 ? 0 : gp == 2 ? 1 : gp == 3 ? 2 : gp == 5 ? 3 : -1;
    if constexpr (wf >= 0) *(f4*)(lw + pb * LBUF + wf * 1152 + (wf >> 1) * 128) = acc[pb][wf];
    if constexpr (gp == 9) asm volatile("s_waitcnt lgkmcnt(0)\n\ts_barrier" ::: "memory");
    auto rd = [&](int part) INL {
      const f2 a = *(const f2*)(lr + pb * LBUF + part * LPART), b = *(const f2*)(lr + pb * LBUF + part * LPART + 1152);
      return (f4){a[0], a[1], b[0], b[1]};
    };
    if constexpr (gp == 11) { s0 = rd(0); s1 = rd(1); }
    if constexpr (gp == 12) { s2 = rd(2); s3 = rd(3); }
    if constexpr (gp == 16) { szO = js >= jmin ? MM * NN * 4 : 0; soS = soO + js * (7 * 16 * NN * 4); PINS(szO); PINS(soS); }
    constexpr int z = gp == 32 ? 8 : gp < 16 || m == 0 || m > 12 || m % 3 == 1 ? -1 : (gp >> 4) * 8 - 8 + (gp > 32) + m - 2 - (m - 2) / 3, e = 2 * (z & 1);
    if constexpr (z == 0 || z == 1) { ADD(s0[e], s0[e], s1[e]); ADD(s0[e + 1], s0[e + 1], s1[e + 1]); }
    if constexpr (z == 2 || z == 3) { ADD(s2[e], s2[e], s3[e]); ADD(s2[e + 1], s2[e + 1], s3[e + 1]); }
    if constexpr (z == 4 || z == 5) { ADD(s0[e], s0[e], s2[e]); ADD(s0[e + 1], s0[e + 1], s2[e + 1]); }
    if constexpr (z == 6 || z == 7) { MUL(s1[e], s0[e], s0[e]); MUL(s1[e + 1], s0[e + 1], s0[e + 1]); }
    if constexpr (z == 8 || z == 9) { FMA(s1[e], s1[e], k1, k0); FMA(s1[e + 1], s1[e + 1], k1, k0); }
    if constexpr (z == 10 || z == 11) { MUL(s1[e], s1[e], s0[e]); MUL(s1[e + 1], s1[e + 1], s0[e + 1]); }
    if constexpr (z >= 12 && z < 16) { s1[z - 12] = __builtin_amdgcn_exp2f(s1[z - 12]); PIN(s1); }
    if constexpr (z == 16 || z == 17) { ADDS(s1[e], one, s1[e]); ADDS(s1[e + 1], one, s1[e + 1]); }
    if constexpr (z >= 18 && z < 22) { s1[z - 18] = __builtin_amdgcn_rcpf(s1[z - 18]); PIN(s1); }
    if constexpr (z == 22 || z == 23) { MUL(s2[e >> 1], s1[e], s0[e]); MUL(s2[(e >> 1) + 2], s1[e + 1], s0[e + 1]); }
    if constexpr (z == 24) {
      __builtin_amdgcn_raw_buffer_store_b128(__builtin_bit_cast(u4, s2), make_rsrc(p.out, szO), voffO, soS, OUTAUX);
    }
  };
#define LD_(d, s, o) "buffer_load_dwordx4 %[d" #d "], %[vo], %[rq], %[" #s "] offen offset:%[" #o "] " LDBITS "\n\t"
#define FIRSTS(n) [a0] "v"(aa[(n) & 7][0][0]), [a1] "v"(aa[(n) & 7][1][0]), [a2] "v"(aa[(n) & 7][2][0]), [a3] "v"(aa[(n) & 7][3][0])
  auto again = [&](auto u_, auto id_) INL {
    constexpr int u = u_.value, id = id_.value;
    (void)aa; (void)rq; (void)so; (void)voffN; (void)left; (void)thr; (void)live; (void)one; (void)tmin;
    asm volatile(".subsection 1\n.Lslow%c0:\n\ts_sleep %1\n\t.subsection 0" :: "i"(id), "i"(XSLEEP));
    sfor<u, u + DD>([&](auto n_) INL {
      constexpr int n = n_.value, qn = n & 3;
      (void)aa; (void)rq; (void)so; (void)voffN;
      asm volatile(".subsection 1\n\t" LD_(0, sa, oa) LD_(1, sa, ob) LD_(2, sb, oa) LD_(3, sb, ob) ".subsection 0"
                   :: [d0] "v"(aa[n & 7][0]), [d1] "v"(aa[n & 7][1]), [d2] "v"(aa[n & 7][2]), [d3] "v"(aa[n & 7][3]),
                      [vo] "v"(voffN[(n >> 2) & 1]), [rq] "s"(rq), [sa] "s"(so[qn >> 1]), [sb] "s"(so[2 + (qn >> 1)]),
                      [oa] "i"((qn & 1) * 2048), [ob] "i"((qn & 1) * 2048 + 1024) : "memory");
    });
    asm volatile(".subsection 1\n\ts_sub_u32 %[left], %[left], 1\n\ts_cbranch_scc0 .Lgo%c[id]\n\t"
                 "s_mov_b32 %[t0], 0\n\ts_mov_b32 %[t1], 0\n\ts_mov_b32 %[live], 0\n\ts_mov_b32 %[one], 0x7fc00000\n.Lgo%c[id]:\n\t"
                 "s_waitcnt vmcnt(%[n])\n\tv_min3_u32 %[t], %[a0], %[a1], %[a2]\n\tv_min_u32 %[t], %[t], %[a3]\n\t"
                 "v_cmp_gt_u32 vcc, %[t0], %[t]\n\ts_cbranch_vccnz .Lslow%c[id]\n\ts_branch .Lback%c[id]\n\t.subsection 0\n\t"
                 "s_cbranch_vccnz .Lslow%c[id]\n.Lback%c[id]:"
                 : [left] "+s"(left), [t0] "+s"(thr[(u >> 2) & 1]), [t1] "+s"(thr[((u >> 2) & 1) ^ 1]), [live] "+s"(live), [one] "+s"(one)
                 : FIRSTS(u), [t] "v"(tmin), [n] "i"(4 * DD - 4), [id] "i"(id) : "memory", "scc", "vcc");
  };

  PINS(thr[0]); PINS(thr[1]);
  int j0 = -4, jend = -2;
#pragma unroll 1
  for (;;) {
    sfor<0, 8>([&](auto st_) INL {
      constexpr int st = st_.value, q = st & 3, pb = st >> 2, n = st + DD, u = st + 1;
      constexpr int qn = n & 3, par = (n >> 2) & 1;
      sfor<0, 16>([&](auto m_) INL {
        constexpr int m = m_.value, k = m >> 2, e = m & 3;
        __builtin_amdgcn_sched_barrier(0);
        acc[pb][e] = MFMA16(bb[k >> 1][2 * q + (k & 1)][e], aa[st][k], (q == 0 && k == 0 ? (f4){0.f, 0.f, 0.f, 0.f} : acc[pb][e]));
        __builtin_amdgcn_sched_barrier(0);
        if constexpr (q == 3 && m == 0) { voffN[pb] = voffD + (j0 + pb + 2) * (7 * 8192); PIN(voffN[pb]); }
        if constexpr (m % 3 == 1 && m <= 10) {
          constexpr int kk = m / 3;
          asm volatile("buffer_load_dwordx4 %0, %1, %2, %3 offen offset:%4 " LDBITS : "=v"(aa[n & 7][kk])
                       : "v"(voffN[par]), "s"(rq), "s"(so[(kk & 2) + (qn >> 1)]), "i"((qn & 1) * 2048 + (kk & 1) * 1024) : "memory");
        }
        if constexpr (q == 0 && m == 6) { thr[pb ^ 1] = (unsigned)(j0 + pb + 1) < 20u ? live : 0u; PINS(thr[pb ^ 1]); }
        if constexpr (m == 13)
          asm volatile("s_waitcnt vmcnt(%[n])\n\tv_min3_u32 %[t], %[a0], %[a1], %[a2]" :: FIRSTS(u), [t] "v"(tmin), [n] "i"(4 * DD - 4) : "memory");
        if constexpr (m == 14)
          asm volatile("v_min_u32 %[t], %[t], %[a3]\n\tv_cmp_gt_u32 vcc, %[t0], %[t]" :: FIRSTS(u), [t] "v"(tmin), [t0] "s"(thr[(u >> 2) & 1]) : "memory", "vcc");
        if constexpr (m == 15) {
          again(IC<u>{}, IC<st>{});
          asm volatile("" : "+v"(aa[u & 7][0]), "+v"(aa[u & 7][1]), "+v"(aa[u & 7][2]), "+v"(aa[u & 7][3]));
        }
        piece(IC<16 * q + m>{}, IC<(pb ^ 1)>{}, j0 + pb - 1);
        __builtin_amdgcn_sched_barrier(0);
      });
    });
    j0 += 2;
    if (RARE(j0 == jend)) {
      sfor<0, 64>([&](auto i_) INL { piece(i_, IC<1>{}, j0 - 1); });
      asm volatile("s_cmp_lt_i32 %0, 0\n\ts_cbranch_scc1 1f\n\ts_endpgm\n1:" :: "s"(j0) : "memory", "scc");
      {
        {
          const __amdgpu_buffer_rsrc_t rz = make_rsrc(myws + DATA_OFF + ((need ^ 1u) & 1u) * PAR_BYTES, PAR_BYTES);
#pragma unroll
          for (int k = 0; k < 11; ++k) {
            const int f = (g * 32 + c) * 4 + v + k * NCW, fw = f - 8192;
            __builtin_amdgcn_raw_buffer_store_b128((u4){0u, 0u, 0u, 0u}, rz, fw < 1024 ? lane * 16 : OOB,
              fw < 0 ? (f >> 10) * SRC_BYTES + (f & 1023) * 1024 : ((fw >> 7) & 7) * SRC_BYTES + W_PART + ((fw >> 2) & 31) * 32768 + 7 * 4096 + (fw & 3) * 1024, AUX_SYS);
          }
        }
        wait_vm<0>();
        KEEP_AA;
        asm volatile("" :: "v"(junk));
        __builtin_amdgcn_s_barrier();
        __builtin_amdgcn_raw_buffer_store_b32(need, make_rsrc(myws, DATA_OFF), lane == 0 && v == 0 ? (int)CLR_OFF : OOB, (g * 32 + c) * 4, AUX_SYS);
        STAMP(15);
        got = 0;
#pragma unroll
        for (int q = 0; q < 4; ++q) {
          if (q < 3) { WAIT_FLAGS(flw, (q ? 0x101u << q : 0x1111u), WSLEEP); } else { WAIT_FLAGS(flw, 0x101u << q, 1); }
          asm volatile("" ::: "memory");
          STAMP(1 + q);
#pragma unroll
          for (int k = 0; k < (q < 3 ? 16 : 8); ++k) {
            const int s = q < 3 ? 2 * q + (k >> 3) : q + 3, r = (k >> 2) & 1;
            if (in_v(r, s, k & 3)) ALD("=v", bb[r][s][k & 3], voffW, r * SRC_BYTES + s * 4096, (k & 3) * 1024);
            else ALD("=a", bb[r][s][k & 3], voffW, r * SRC_BYTES + s * 4096, (k & 3) * 1024);
          }
        }
        for (;;) {
#pragma unroll
          for (int k = 0; k < 8; ++k) ALD("=v", bb[k >> 2][7][k & 3], voffW, (k >> 2) * SRC_BYTES + 7 * 4096, (k & 3) * 1024);
          asm volatile("s_waitcnt vmcnt(0)" : "+v"(bb[0][7][0]), "+v"(bb[0][7][1]), "+v"(bb[0][7][2]), "+v"(bb[0][7][3]),
                                              "+v"(bb[1][7][0]), "+v"(bb[1][7][1]), "+v"(bb[1][7][2]), "+v"(bb[1][7][3]) :: "memory");
          unsigned lo = bb[0][7][0][0];
#pragma unroll
          for (int k = 1; k < 8; ++k) lo = min(lo, bb[k >> 2][7][k & 3][0]);
          if (__ballot(lo == 0u) == 0ull || need == 0u) break;
          __builtin_amdgcn_s_sleep(W7SLEEP);
          if (++polls > MAX_POLLS) need = 0u;
        }
        STAMP(7);
        one = need ? one : 0x7fc00000u;
        asm volatile("s_min_u32 %0, %1, 1" : "=s"(live) : "s"(need) : "scc");
        j0 = jmin & 2;
        voffD = voffX;
        voffN[0] = jmin & 1 ? OOB : voffX + j0 * 7 * 8192;
        voffN[1] = voffX + (j0 + 1) * 7 * 8192;
        thr[0] = (jmin >> 1) & live;
        PINS(live); PINS(one); PINS(thr[0]); PIN(voffN[0]); PIN(voffN[1]);
        asm volatile("s_mov_b64 vcc, -1" ::: "vcc");
        again(IC<0>{}, IC<8>{});
        jend = 20;
        PINS(jend);
      }
    }
    asm volatile("" ::: "memory");
  }
}

__global__ void __launch_bounds__(256, 1) gemm_ag_gelu(Params p) {
  __shared__ __attribute__((aligned(16))) char lds[2 * LBUF];
  const int lane = threadIdx.x & 63;
  const int wave = __builtin_amdgcn_readfirstlane(threadIdx.x >> 6);
  const int b = blockIdx.x;
  STAMP(0);
  if (__builtin_expect(b < NPUSH, 1)) {
    if (wave < 3) pusher(p, b, wave, lane);
  } else {
    const int i = (b & 7) * 28 + (b >> 3) - NPUSH / 8;
    consumer(p, i >> 5, i & 31, wave, lane, lds);
  }
}

extern "C" void kernel_launch(int rank, int nranks, ncclComm_t comm,
                              hipStream_t stream,
                              void* const* d_in, const int* in_sizes, int n_in,
                              void* d_out, int out_size,
                              const int* in_dtypes, int out_dtype,
                              void* const* d_out_all,
                              void* const* d_ws_all, size_t ws_bytes) {
  if (nranks != NR || (unsigned)rank >= (unsigned)NR || n_in < 2 || ws_bytes < WS_NEED ||
      in_sizes[0] != MM * KS || in_sizes[1] != KS * NN || out_size != MM * NN) {
    fprintf(stderr, "kernel_launch: unexpected problem shape\n");
    return;
  }
  Params p;
  p.x = (const float*)d_in[0];
  p.w = (const float*)d_in[1];
  p.out = (float*)d_out;
  for (int q = 0; q < 2 * NR; ++q) p.ws2[q] = (char*)d_ws_all[q & (NR - 1)] + DATA_OFF;
  p.myws = (char*)d_ws_all[rank];
  p.rank = rank;
  gemm_ag_gelu<<<256, 256, 0, stream>>>(p);
  HIP_CHECK(hipGetLastError());
}
```
